```python
import numpy as np

kernel_name = "gpu_gemm_a2a_m2048_k2048_n2048_bf16_none"


ABSMAX_RTOL = 0.008

COMPUTE_DTYPE = "bfloat16"
EPILOGUE = "none"
IN_QUANT = None
OUT_QUANT = None
IN_DTYPES = {"x": "bfloat16", "w_mat": "bfloat16"}
WORLD_SIZE = 8

def _to_bf16(x):
    u = np.ascontiguousarray(x, dtype=np.float32).view(np.uint32)
    return ((u + 0x7FFF + ((u >> 16) & 1)) & 0xFFFF0000).view(np.float32)

def setup_inputs():

    x = _to_bf16(np.random.randn(2048, 2048).astype(np.float32))
    w_mat = _to_bf16((np.random.randn(2048, 2048) / np.sqrt(2048)).astype(np.float32))
    return {"x": x, "w_mat": w_mat}

def _epilogue(y):
    return y

def reference(x, w_mat):
    y = (x @ w_mat).astype(np.float32)
    return _epilogue(y)
```

```cpp
#include <hip/hip_runtime.h>
#include <rccl/rccl.h>
#include <cstdio>

#define HIP_CHECK(expr) do { hipError_t _e = (expr); if (_e != hipSuccess) { \
    fprintf(stderr, "HIP error %s at %s:%d\n", hipGetErrorString(_e), __FILE__, __LINE__); \
    return; } } while (0)

#define WT_STORES 0

typedef unsigned int u32;
typedef u32 u32x2 __attribute__((ext_vector_type(2)));
typedef u32 u32x4 __attribute__((ext_vector_type(4)));
typedef float f32x4 __attribute__((ext_vector_type(4)));
typedef __bf16 bf16x8 __attribute__((ext_vector_type(8)));
typedef __bf16 bf16x2 __attribute__((ext_vector_type(2)));
typedef float f32x2 __attribute__((ext_vector_type(2)));

constexpr int NR = 8;
constexpr int MLOC = 256;
constexpr int KD = 2048;
constexpr int ND = 2048;
constexpr int NLOC = 256;
constexpr int NWG = 256;
constexpr int NTHR = 512;
constexpr size_t WS_FLAGS = 0;
constexpr size_t WS_CNT = 4096;
constexpr size_t WS_TMO = 8192;
constexpr size_t WS_NEED = 16384;
constexpr unsigned long long DEADLINE_TICKS = 50000000ull;

#define GAS __attribute__((address_space(1)))
struct Params {
    const void* x;
    const void* w;
    void* myws;
    void* out[NR];
    void* ws[NR];
    int rank;
};

__device__ __forceinline__ void glds16(const char* sbase, u32 voff, u32 lds_dst) {
    asm volatile("s_mov_b32 m0, %2\n\ts_nop 0\n\tglobal_load_lds_dwordx4 %0, %1"
                 :: "v"(voff), "s"(sbase), "s"(lds_dst) : "memory");
}
__device__ __forceinline__ void gload8(u32x2& d, const char* sbase, u32 voff) {
    asm volatile("global_load_dwordx2 %0, %1, %2" : "=v"(d) : "v"(voff), "s"(sbase));
}
__device__ __forceinline__ u32 pack_bf16(float a, float b) {
    f32x2 v = {a, b};
    bf16x2 r = __builtin_convertvector(v, bf16x2);
    return __builtin_bit_cast(u32, r);
}

__global__ void __launch_bounds__(NTHR, 2) gemm_a2a(Params p) {
    __shared__ __attribute__((aligned(1024))) char smem[128 * 1024];
    const int b = blockIdx.x;
    const int g = b & 7;
    const int i = b >> 3;
    const int dst = g;
    const int m0 = (i & 7) * 32;
    const int nl0 = (i >> 3) * 64;
    const int n0 = dst * NLOC + nl0;
    const int tid = threadIdx.x;
    const int lane = tid & 63;
    const int w = __builtin_amdgcn_readfirstlane(tid >> 6);
    const int rank = p.rank;

    GAS char* myws = (GAS char*)p.myws;
    GAS char* dws = (GAS char*)p.ws[dst];
    GAS char* dout = (GAS char*)p.out[dst];
    asm volatile("" : "+s"(myws), "+s"(dws), "+s"(dout));

    const u32 lds0 = (u32)(size_t)smem + (u32)w * 16384u;
    {
        const int r = lane >> 3, c = lane & 7;
        const char* xb = (const char*)p.x + (size_t)m0 * (KD * 2) + (size_t)w * 512;
        const u32 v0 = (u32)(r * (KD * 2) + ((c ^ (r >> 1)) * 16));
        const u32 v1 = (u32)(r * (KD * 2) + ((c ^ (4 + (r >> 1))) * 16));
#pragma unroll
        for (int kb = 0; kb < 4; ++kb)
#pragma unroll
            for (int rg = 0; rg < 4; ++rg)
                glds16(xb + rg * 8 * (KD * 2) + kb * 128, (rg & 1) ? v1 : v0, lds0 + (kb * 4 + rg) * 1024);
    }
    const int c16 = lane & 15, q = lane >> 4;
    u32x2 wr[8][8];
    {
        const char* wb = (const char*)p.w + (size_t)w * 256 * (ND * 2) + (size_t)n0 * 2;
        const u32 vo = (u32)(q * 8 * (ND * 2) + c16 * 8);
#pragma unroll
        for (int ks = 0; ks < 8; ++ks)
#pragma unroll
            for (int j = 0; j < 8; ++j)
                gload8(wr[ks][j], wb + (ks * 32 + j) * (ND * 2), vo);
    }
    f32x4 acc[2][4];
#pragma unroll
    for (int mi = 0; mi < 2; ++mi)
#pragma unroll
        for (int ni = 0; ni < 4; ++ni) acc[mi][ni] = (f32x4){0.f, 0.f, 0.f, 0.f};

    const char* xa = smem + w * 16384 + (c16 >> 3) * 1024 + (c16 & 7) * 128;
    const int xs0 = ((0 + q) ^ (c16 >> 1)) * 16, xs1 = ((4 + q) ^ (c16 >> 1)) * 16;

#define WAITW(ks, N, ...)                                                                     \
    asm volatile("s_waitcnt vmcnt(" #N ")"                                                    \
                 : "+v"(wr[ks][0]), "+v"(wr[ks][1]), "+v"(wr[ks][2]), "+v"(wr[ks][3]),        \
                   "+v"(wr[ks][4]), "+v"(wr[ks][5]), "+v"(wr[ks][6]), "+v"(wr[ks][7]) :: __VA_ARGS__)
#define STEP(ks)                                                                              \
    {                                                                                         \
        bf16x8 a0 = *(const bf16x8*)(xa + (ks >> 1) * 4096 + ((ks & 1) ? xs1 : xs0));         \
        bf16x8 a1 = *(const bf16x8*)(xa + (ks >> 1) * 4096 + 2048 + ((ks & 1) ? xs1 : xs0));  \
        _Pragma("unroll") for (int ni = 0; ni < 4; ++ni) {                                    \
            u32x4 bf;                                                                         \
            _Pragma("unroll") for (int d = 0; d < 4; ++d)                                     \
                bf[d] = __builtin_amdgcn_perm(wr[ks][2 * d + 1][ni >> 1], wr[ks][2 * d][ni >> 1], \
                                              (ni & 1) ? 0x07060302u : 0x05040100u);          \
            bf16x8 bb = __builtin_bit_cast(bf16x8, bf);                                       \
            acc[0][ni] = __builtin_amdgcn_mfma_f32_16x16x32_bf16(a0, bb, acc[0][ni], 0, 0, 0); \
            acc[1][ni] = __builtin_amdgcn_mfma_f32_16x16x32_bf16(a1, bb, acc[1][ni], 0, 0, 0); \
        }                                                                                     \
    }
    WAITW(0, 56, "memory"); STEP(0)
    WAITW(1, 48); STEP(1)
    WAITW(2, 40); STEP(2)
    WAITW(3, 32); STEP(3)
    WAITW(4, 24); STEP(4)
    WAITW(5, 16); STEP(5)
    WAITW(6, 8); STEP(6)
    WAITW(7, 0); STEP(7)

    u32 call;
    asm volatile("global_load_dword %0, %1, %2 sc1" : "=v"(call) : "v"((u32)(WS_CNT + b * 4)), "s"(myws) : "memory");
    {
        char* pw = smem + w * 16384 + q * 4 * 256 + c16 * 16;
#pragma unroll
        for (int mi = 0; mi < 2; ++mi)
#pragma unroll
            for (int r = 0; r < 4; ++r) {
                f32x4 v = {acc[mi][0][r], acc[mi][1][r], acc[mi][2][r], acc[mi][3][r]};
                *(f32x4*)(pw + (mi * 16 + r) * 256) = v;
            }
    }
    __syncthreads();
    {
        const char* pr = smem + tid * 16;
        f32x4 s = *(const f32x4*)pr;
#pragma unroll
        for (int k = 1; k < 8; ++k) s += *(const f32x4*)(pr + k * 16384);
        u32 lo = pack_bf16(s[0], s[1]), hi = pack_bf16(s[2], s[3]);
        u32 lo2 = __builtin_amdgcn_update_dpp(0u, lo, 0xB1, 0xF, 0xF, true);
        u32 hi2 = __builtin_amdgcn_update_dpp(0u, hi, 0xB1, 0xF, 0xF, true);
        if ((tid & 1) == 0) {
            u32x4 o = {lo, hi, lo2, hi2};
            GAS char* op = dout + ((size_t)(rank * MLOC + m0 + (tid >> 4)) * NLOC + nl0 + (tid & 15) * 4) * 2;
            *(GAS u32x4*)op = o;
        }
    }
    asm volatile("s_waitcnt vmcnt(0)" : "+v"(call) :: "memory");
    call += 1;
    if (dst != rank) {
        __syncthreads();
        if (tid == 0) {
            __builtin_amdgcn_fence(__ATOMIC_RELEASE, "");
            asm volatile("s_waitcnt vmcnt(0)" ::: "memory");
            __hip_atomic_store((GAS u32*)(dws + WS_FLAGS) + rank * 32 + i, call,
                               __ATOMIC_RELAXED, __HIP_MEMORY_SCOPE_SYSTEM);
        }
    }
    if (tid == 0)
        __hip_atomic_store((GAS u32*)(myws + WS_CNT) + b, call, __ATOMIC_RELAXED, __HIP_MEMORY_SCOPE_AGENT);

    if (b == rank && w == 0) {
        call = __builtin_amdgcn_readfirstlane(call);
        GAS const char* fp = myws + WS_FLAGS + lane * 16;
        const bool skip = (lane >> 3) == rank;
        const unsigned long long t0 = __builtin_amdgcn_s_memrealtime();
        for (;;) {
            u32x4 f;
            asm volatile("global_load_dwordx4 %0, %1, off sc0 sc1\n\ts_waitcnt vmcnt(0)" : "=&v"(f) : "v"(fp) : "memory");
            bool ok = skip || (f[0] >= call && f[1] >= call && f[2] >= call && f[3] >= call);
            if (__all(ok)) break;
            if (__builtin_amdgcn_s_memrealtime() - t0 > DEADLINE_TICKS) {
                if (lane == 0) {
                    __hip_atomic_store((GAS u32*)(myws + WS_TMO), call, __ATOMIC_RELAXED, __HIP_MEMORY_SCOPE_AGENT);
                    ((GAS unsigned short*)dout)[(size_t)rank * MLOC * NLOC] = 0x7FC0;
                }
                break;
            }
            __builtin_amdgcn_s_sleep(1);
        }
    }
}

extern "C" void kernel_launch(int rank, int nranks, ncclComm_t comm,
                              hipStream_t stream,
                              void* const* d_in, const int* in_sizes, int n_in,
                              void* d_out, int out_size,
                              const int* in_dtypes, int out_dtype,
                              void* const* d_out_all,
                              void* const* d_ws_all, size_t ws_bytes) {
    if (nranks != NR || (unsigned)rank >= (unsigned)NR || n_in < 2 || ws_bytes < WS_NEED ||
        in_sizes[0] != MLOC * KD || in_sizes[1] != KD * ND || out_size != NR * MLOC * NLOC) {
        fprintf(stderr, "kernel_launch: unexpected problem shape\n");
        return;
    }
    Params p;
    p.x = d_in[0];
    p.w = d_in[1];
    for (int r = 0; r < NR; ++r) { p.out[r] = d_out_all[r]; p.ws[r] = d_ws_all[r]; }
    p.rank = rank;
    p.myws = d_ws_all[rank];
    hipLaunchKernelGGL(gemm_a2a, dim3(NWG), dim3(NTHR), 0, stream, p);
    HIP_CHECK(hipGetLastError());
}
```

```cpp
#include <hip/hip_runtime.h>
#include <rccl/rccl.h>
#include <cstdio>

#define HIP_CHECK(expr) do { hipError_t _e = (expr); if (_e != hipSuccess) { \
    fprintf(stderr, "HIP error %s at %s:%d\n", hipGetErrorString(_e), __FILE__, __LINE__); \
    return; } } while (0)

#define SB 2
#define RS 3
#define LEAD 1
#define TOUCH 1
#define TDELAY 7
#define ROT (4 * g)

typedef unsigned int u32;
typedef u32 u32x2 __attribute__((ext_vector_type(2)));
typedef u32 u32x4 __attribute__((ext_vector_type(4)));
typedef float f32x4 __attribute__((ext_vector_type(4)));
typedef __bf16 bf16x8 __attribute__((ext_vector_type(8)));
typedef __bf16 bf16x2 __attribute__((ext_vector_type(2)));
typedef float f32x2 __attribute__((ext_vector_type(2)));
typedef short s16x4 __attribute__((ext_vector_type(4)));

constexpr int NR = 8;
constexpr int MLOC = 256;
constexpr int KD = 2048;
constexpr int ND = 2048;
constexpr int NLOC = 256;
constexpr int NWG = 256;
constexpr int NTHR = 512;
constexpr int BLKB = 12288;
constexpr int STB = SB * BLKB;
constexpr int NST = 32 / SB;
constexpr int DD = RS - 1;
constexpr size_t WS_FLAGS = 0;
constexpr size_t WS_CNT = 4096;
constexpr size_t WS_TMO = 8192;
constexpr size_t WS_NEED = 16384;
constexpr unsigned long long DEADLINE_TICKS = 50000000ull;

#define GAS __attribute__((address_space(1)))
#define LAS __attribute__((address_space(3)))
struct Params {
    const void* x;
    const void* w;
    void* myws;
    void* out[NR];
    void* ws[NR];
    int rank;
};
#define WSTAMP(k)
#define WDUMP()

__device__ __forceinline__ u32 pack_bf16(float a, float b) {
    f32x2 v = {a, b};
    bf16x2 r = __builtin_convertvector(v, bf16x2);
    return __builtin_bit_cast(u32, r);
}
#define BARRIER() do { asm volatile("" ::: "memory"); __builtin_amdgcn_s_barrier(); asm volatile("" ::: "memory"); } while (0)
template <int N> __device__ __forceinline__ void wait_vm() { asm volatile("s_waitcnt vmcnt(%0)" :: "n"(N) : "memory"); }

__global__ void __launch_bounds__(NTHR, 2) gemm_a2a(Params p) {
    __shared__ __attribute__((aligned(1024))) char smem[RS * STB > 84 * 1024 ? RS * STB : 84 * 1024];
    const int b = blockIdx.x;
    const int g = b & 7;
    const int i = b >> 3;
#define MAPV 0
    const int mb = i & 7, nb = MAPV == 1 ? 3 - (i >> 3) : MAPV == 2 ? ((i >> 3) ^ 1) : (i >> 3);
    const int dst = MAPV == 3 ? (g ^ 1) : g;
    const int m0 = mb * 32;
    const int nl0 = nb * 64;
    const int tid = threadIdx.x;
    const int lane = tid & 63;
    const int w = __builtin_amdgcn_readfirstlane(tid >> 6);
    WSTAMP(0);
    const char* px = (const char*)p.x;
    const char* pw = (const char*)p.w;
    asm volatile("" : "+s"(px), "+s"(pw));
    const char* xb = px + (size_t)m0 * (KD * 2);
    const u32 lds0 = (u32)(size_t)smem;
    const bool lead = LEAD && (((mb + (i >> 3)) & 1) == 0);
    const int jc = ((mb >> 1) + (nb >> 1)) & 3;
    const int rot = ROT;

    if (__builtin_expect(w < 4, 1)) {
        const char* wb = pw + (size_t)(dst * NLOC + nl0) * 2 - 1024;
        const int r8 = lane >> 3, c8 = lane & 7;
        const u32 vx = (u32)((8 * w + r8) * (KD * 2) + ((c8 ^ ((w & 1) * 4 + (r8 >> 1))) * 16));
        const u32 vwa = (u32)((16 * w + r8) * (ND * 2) + ((c8 ^ (r8 & 2)) * 16));
        const u32 vwb = (u32)((16 * w + 8 + r8) * (ND * 2) + ((c8 ^ ((r8 & 2) | 4)) * 16)) - 1024u;
        int v = 0;
        const int vstep = lead ? 4 : 1, voff = (lead ? jc : 0) + rot;
        u32 ldst = lds0 + w * 3072;
        auto issue_stage = [&]() {
#pragma unroll
            for (int k = 0; k < SB; ++k) {
                const u32 blk = (u32)(v + voff) & 31u;
                const char* sx = xb + blk * 128;
                const char* sw = wb + (size_t)(blk * (64 * ND * 2));
                asm volatile("s_mov_b32 m0, %5\n\ts_nop 0\n\t"
                             "global_load_lds_dwordx4 %0, %3\n\t"
                             "global_load_lds_dwordx4 %1, %4 offset:1024\n\t"
                             "global_load_lds_dwordx4 %2, %4 offset:2048"
                             :: "v"(vx), "v"(vwa), "v"(vwb), "s"(sx), "s"(sw), "s"(ldst) : "memory");
                v += vstep;
                v = v >= 32 ? v - 31 : v;
                ldst += BLKB;
            }
            if (ldst >= lds0 + RS * STB) ldst -= RS * STB;
        };
        WSTAMP(1);
#pragma unroll 1
        for (int s = 0; s < DD; ++s) issue_stage();
        WSTAMP(2);
#pragma unroll 1
        for (int s = 0; s < NST - DD; ++s) {
            wait_vm<3 * SB * (DD - 1)>();
            BARRIER();
            issue_stage();
        }
        WSTAMP(3);
        if (DD > 5) { wait_vm<(DD > 5 ? 3 * SB * 5 : 0)>(); BARRIER(); }
        if (DD > 4) { wait_vm<(DD > 4 ? 3 * SB * 4 : 0)>(); BARRIER(); }
        if (DD > 3) { wait_vm<(DD > 3 ? 3 * SB * 3 : 0)>(); BARRIER(); }
        if (DD > 2) { wait_vm<(DD > 2 ? 3 * SB * 2 : 0)>(); BARRIER(); }
        if (DD > 1) { wait_vm<(DD > 1 ? 3 * SB * 1 : 0)>(); BARRIER(); }
        wait_vm<0>(); BARRIER();
        WSTAMP(4);
        if (w != 0) { WDUMP(); return; }
        const int rank = p.rank;
        GAS char* myws = (GAS char*)p.myws;
        GAS char* dws = (GAS char*)p.ws[dst];
        u32 call = __hip_atomic_load((GAS u32*)(myws + WS_CNT) + b, __ATOMIC_RELAXED, __HIP_MEMORY_SCOPE_AGENT) + 1;
        call = __builtin_amdgcn_readfirstlane(call);
        if (lane == 0) __hip_atomic_store((GAS u32*)(myws + WS_CNT) + b, call, __ATOMIC_RELAXED, __HIP_MEMORY_SCOPE_AGENT);
        BARRIER();
        WSTAMP(5);
        if (dst != rank) {
            if (lane == 0)
                __hip_atomic_store((GAS u32*)(dws + WS_FLAGS) + rank * 32 + i, call, __ATOMIC_RELAXED, __HIP_MEMORY_SCOPE_SYSTEM);
        } else if (i == 0) {
            GAS const char* fp = myws + WS_FLAGS + lane * 16;
            const bool skip = (lane >> 3) == rank;
            const unsigned long long t0 = __builtin_amdgcn_s_memrealtime();
            for (;;) {
                u32x4 f;
                asm volatile("global_load_dwordx4 %0, %1, off sc0 sc1\n\ts_waitcnt vmcnt(0)" : "=&v"(f) : "v"(fp) : "memory");
                bool ok = skip || (f[0] >= call && f[1] >= call && f[2] >= call && f[3] >= call);
                if (__all(ok)) break;
                if (__builtin_amdgcn_s_memrealtime() - t0 > DEADLINE_TICKS) {
                    if (lane == 0) {
                        __hip_atomic_store((GAS u32*)(myws + WS_TMO), call, __ATOMIC_RELAXED, __HIP_MEMORY_SCOPE_AGENT);
                        ((GAS unsigned short*)p.out[dst])[(size_t)rank * MLOC * NLOC] = 0x7FC0;
                    }
                    break;
                }
                __builtin_amdgcn_s_sleep(1);
            }
        }
        WSTAMP(6);
        WDUMP();
        return;
    }

    const int c = w - 4, mi = c & 1, nh = c >> 1;
    u32 touched = 0;
    if (TOUCH && lead) {
        for (int k = 0; k < c; ++k) __builtin_amdgcn_s_sleep(TDELAY);
        const int bb = (jc + 2 + ((2 * c + (lane >> 5)) << 2) + rot) & 31;
        asm volatile("global_load_dword %0, %1, %2" : "=v"(touched) : "v"((u32)((lane & 31) * (KD * 2) + bb * 128)), "s"(xb) : "memory");
    }
    GAS char* op = (GAS char*)p.out[dst] + ((size_t)(p.rank * MLOC + m0 + mi * 16 + (lane >> 2)) * NLOC + nl0 + nh * 32 + (lane & 3) * 8) * 2;
    const int c16 = lane & 15, q = lane >> 4;
    u32 xa[2], wa[2];
#pragma unroll
    for (int s = 0; s < 2; ++s) xa[s] = lds0 + (2 * mi + (c16 >> 3)) * 3072 + (c16 & 7) * 128 + (((4 * s + q) ^ (c16 >> 1)) * 16);
    {
        const int qq = c16 >> 2, pp = c16 & 3, x = ((qq >> 1) & 1) | ((q & 1) << 1);
#pragma unroll
        for (int t = 0; t < 2; ++t) wa[t] = lds0 + (q >> 1) * 3072 + 1024 + (q & 1) * 1024 + qq * 128 + (((2 * nh + t) ^ x) * 32) + pp * 8;
    }
    const u32 eb = lds0 + ((NST - 2) % RS) * STB + c * 1024;
    u32 ew0 = eb + c16 * 64 + (((q >> 1) ^ (c16 >> 1)) & 3) * 16 + (q & 1) * 8;
    u32 er = eb + (lane >> 2) * 64 + (((lane & 3) ^ (lane >> 3)) & 3) * 16;
    asm volatile("s_waitcnt lgkmcnt(0)" : "+v"(op), "+v"(ew0), "+v"(er), "+v"(xa[0]), "+v"(xa[1]), "+v"(wa[0]), "+v"(wa[1]));
    f32x4 acc[2] = {{0.f, 0.f, 0.f, 0.f}, {0.f, 0.f, 0.f, 0.f}};
    int slot = 0;
    WSTAMP(1);
#define RDSTEP(u)                                                                                          \
    asm volatile("ds_read_b128 %0, %5 offset:%8\n\t"                                                       \
                 "ds_read_b64_tr_b16 %1, %6 offset:%9\n\tds_read_b64_tr_b16 %2, %6 offset:%10\n\t"         \
                 "ds_read_b64_tr_b16 %3, %7 offset:%9\n\tds_read_b64_tr_b16 %4, %7 offset:%10"             \
                 : "=&v"(xf[u]), "=&v"(wf[u][0]), "=&v"(wf[u][1]), "=&v"(wf[u][2]), "=&v"(wf[u][3])        \
                 : "v"(xa[(u) & 1]), "v"(wa[0]), "v"(wa[1]), "i"(((u) >> 1) * BLKB),                       \
                   "i"(((u) >> 1) * BLKB + ((u) & 1) * 6144), "i"(((u) >> 1) * BLKB + ((u) & 1) * 6144 + 512) : "memory")
#define MMASTEP(u, N)                                                                                      \
    do {                                                                                                   \
        asm volatile("s_waitcnt lgkmcnt(" #N ")" : "+v"(xf[u]), "+v"(wf[u][0]), "+v"(wf[u][1]), "+v"(wf[u][2]), "+v"(wf[u][3]) :: "memory"); \
        u32x4 t0 = {wf[u][0][0], wf[u][0][1], wf[u][1][0], wf[u][1][1]};                                   \
        u32x4 t1 = {wf[u][2][0], wf[u][2][1], wf[u][3][0], wf[u][3][1]};                                   \
        acc[0] = __builtin_amdgcn_mfma_f32_16x16x32_bf16(__builtin_bit_cast(bf16x8, t0), __builtin_bit_cast(bf16x8, xf[u]), acc[0], 0, 0, 0); \
        acc[1] = __builtin_amdgcn_mfma_f32_16x16x32_bf16(__builtin_bit_cast(bf16x8, t1), __builtin_bit_cast(bf16x8, xf[u]), acc[1], 0, 0, 0); \
        __builtin_amdgcn_sched_barrier(0);                                                                 \
    } while (0)
    {
#pragma unroll 1
    for (int st = 0; st < NST; ++st) {
        u32x4 xf[4];
        u32x2 wf[4][4];
        BARRIER();
        slot = slot == RS - 1 ? 0 : slot + 1;
        const int delta = slot == 0 ? -(RS - 1) * STB : STB;
        RDSTEP(0); RDSTEP(1); RDSTEP(2);
        MMASTEP(0, 10);
        RDSTEP(3);
        xa[0] += delta; xa[1] += delta; wa[0] += delta; wa[1] += delta;
        MMASTEP(1, 10);
        MMASTEP(2, 5);
        MMASTEP(3, 0);
    }
    }
    WSTAMP(7);
    {
        u32x2 v0 = {pack_bf16(acc[0][0], acc[0][1]), pack_bf16(acc[0][2], acc[0][3])};
        u32x2 v1 = {pack_bf16(acc[1][0], acc[1][1]), pack_bf16(acc[1][2], acc[1][3])};
        *(LAS u32x2*)(size_t)ew0 = v0;
        *(LAS u32x2*)(size_t)(ew0 ^ 32) = v1;
        const u32x4 o = *(const LAS u32x4*)(size_t)er;
        asm volatile("global_store_dwordx4 %0, %1, off sc0 sc1\n\ts_nop 1" :: "v"(op), "v"(o) : "memory");
    }
    WSTAMP(8);
    asm volatile("s_waitcnt vmcnt(0)" : "+v"(touched) :: "memory");
    WSTAMP(9);
    BARRIER();
    WDUMP();
}

extern "C" void kernel_launch(int rank, int nranks, ncclComm_t comm,
                              hipStream_t stream,
                              void* const* d_in, const int* in_sizes, int n_in,
                              void* d_out, int out_size,
                              const int* in_dtypes, int out_dtype,
                              void* const* d_out_all,
                              void* const* d_ws_all, size_t ws_bytes) {
    if (nranks != NR || (unsigned)rank >= (unsigned)NR || n_in < 2 || ws_bytes < WS_NEED ||
        in_sizes[0] != MLOC * KD || in_sizes[1] != KD * ND || out_size != NR * MLOC * NLOC) {
        fprintf(stderr, "kernel_launch: unexpected problem shape\n");
        return;
    }
    Params p;
    p.x = d_in[0];
    p.w = d_in[1];
    for (int r = 0; r < NR; ++r) { p.out[r] = d_out_all[r]; p.ws[r] = d_ws_all[r]; }
    p.rank = rank;
    p.myws = d_ws_all[rank];
    hipLaunchKernelGGL(gemm_a2a, dim3(NWG), dim3(NTHR), 0, stream, p);
    HIP_CHECK(hipGetLastError());
}
```

```cpp
#include <hip/hip_runtime.h>
#include <rccl/rccl.h>
#include <cstdio>

#define HIP_CHECK(expr) do { hipError_t _e = (expr); if (_e != hipSuccess) { \
    fprintf(stderr, "HIP error %s at %s:%d\n", hipGetErrorString(_e), __FILE__, __LINE__); \
    return; } } while (0)

#define BPB 1
#define XTRA 0
#define LEAD 1
#define TDELAY 7
#define ROT (4 * g)
#define C3FIX 1

typedef unsigned int u32;
typedef u32 u32x2 __attribute__((ext_vector_type(2)));
typedef u32 u32x4 __attribute__((ext_vector_type(4)));
typedef u32 u32x8 __attribute__((ext_vector_type(8)));
typedef float f32x4 __attribute__((ext_vector_type(4)));
typedef __bf16 bf16x8 __attribute__((ext_vector_type(8)));
typedef __bf16 bf16x2 __attribute__((ext_vector_type(2)));
typedef float f32x2 __attribute__((ext_vector_type(2)));

constexpr int NR = 8;
constexpr int MLOC = 256;
constexpr int KD = 2048;
constexpr int ND = 2048;
constexpr int NLOC = 256;
constexpr int NWG = 256;
constexpr int NTHR = 512;
constexpr int BLKB = 12288;
constexpr int RINGB = 8 * BLKB;
constexpr int PB = 4 + XTRA;
constexpr size_t WS_FLAGS = 0;
constexpr size_t WS_CNT = 4096;
constexpr size_t WS_TMO = 8192;
constexpr size_t WS_NEED = 16384;
constexpr unsigned long long DEADLINE_TICKS = 50000000ull;

#define GAS __attribute__((address_space(1)))
#define LAS __attribute__((address_space(3)))
struct Params {
    u32 rx[4];
    u32 rw[4];
    void* myws;
    int rank, pad;
    void* out[NR];
    void* ws[NR];
};
#define WSTAMP(k)
#define WDUMP()

__device__ __forceinline__ u32 pack_bf16(float a, float b) {
    f32x2 v = {a, b};
    bf16x2 r = __builtin_convertvector(v, bf16x2);
    return __builtin_bit_cast(u32, r);
}
#define GLDS4(sbase, voff, lds_dst, bits) \
    asm volatile("s_mov_b32 m0, %2\n\ts_nop 0\n\tglobal_load_lds_dword %0, %1 " bits :: "v"(voff), "s"(sbase), "s"(lds_dst) : "memory")
#define BARRIER() do { asm volatile("" ::: "memory"); __builtin_amdgcn_s_barrier(); asm volatile("" ::: "memory"); } while (0)
template <int N> __device__ __forceinline__ void wait_vm() { asm volatile("s_waitcnt vmcnt(%0)" :: "n"(N) : "memory"); }

__global__ void __launch_bounds__(NTHR, 2) gemm_a2a(Params p) {
    __shared__ __attribute__((aligned(1024))) char smem[RINGB + 1024];
    u32x8 kxw;
    asm volatile("s_load_dwordx8 %0, %1, 0x0" : "=s"(kxw) : "s"(__builtin_amdgcn_kernarg_segment_ptr()));
    __builtin_amdgcn_sched_barrier(0);
    const int b = blockIdx.x;
    const int g = b & 7;
    const int i = b >> 3;
    const int mb = i & 7, nb = i >> 3;
    const int dst = (g + 2 * nb) & 7;
    const int m0 = mb * 32;
    const int nl0 = nb * 64;
    const int tid = threadIdx.x;
    const int lane = tid & 63;
    const int w = __builtin_amdgcn_readfirstlane(tid >> 6);
    WSTAMP(0);
    const u32 lds0 = (u32)(size_t)smem;
    const bool lead = LEAD && (((mb + nb) & 1) == 0);
    const int jc = ((mb >> 1) + (nb >> 1)) & 3;
    const int rot = ROT;

    const int r8 = lane >> 3, c8 = lane & 7;
    const u32 vx = (u32)((m0 + 8 * w + r8) * (KD * 2) + ((c8 ^ ((w & 1) * 4 + (r8 >> 1))) * 16));
    const u32 vwa = (u32)((16 * w + r8) * (ND * 2) + (dst * NLOC + nl0) * 2 + ((c8 ^ (r8 & 2)) * 16));
    const u32 vwb = (vwa ^ 64u) + (8 * ND * 2 - 1024);
    u32 sox = (u32)((((lead ? jc : 0) + rot) & 31) * 128);
    const u32 step = lead ? 512u : 128u, adjv = lead ? 128u : 0u;
    u32 lb = lds0 + w * 3072;
    asm volatile("s_waitcnt lgkmcnt(0)" : "+s"(kxw) : "s"(sox), "s"(lb), "v"(vx), "v"(vwa), "v"(vwb), "s"(step));
    const u32x4 rx = {kxw[0], kxw[1], kxw[2], kxw[3]};
    const u32x4 rw = {kxw[4], kxw[5], kxw[6], kxw[7]};

    if (__builtin_expect(w < 4, 1)) {
        u32 adj = adjv;
#define ISSUE(pre, slot)                                                                           \
        { u32 sow;                                                                                 \
        asm volatile("s_lshl_b32 %1, %0, 11\n\t"                                                   \
                     "s_add_u32 m0, %7, %9\n\t"                                                    \
                     pre                                                                           \
                     "buffer_load_dwordx4 %2, %5, %0 offen lds\n\t"                                \
                     "buffer_load_dwordx4 %3, %6, %1 offen offset:1024 lds\n\t"                    \
                     "buffer_load_dwordx4 %4, %6, %1 offen offset:2048 lds\n\t"                    \
                     "s_add_u32 %0, %0, %8\n\t"                                                    \
                     "s_and_b32 %0, %0, 0xf80"                                                     \
                     : "+s"(sox), "=&s"(sow) : "v"(vx), "v"(vwa), "v"(vwb), "s"(rx), "s"(rw), "s"(lb), "s"(step), "i"((slot) * BLKB) : "memory", "scc"); }
#define STR2(x) #x
#define STR(x) STR2(x)
#define NOSYNC "s_nop 0\n\t"
#define SYNC_A "s_waitcnt vmcnt(9)\n\ts_barrier\n\t"
#define SYNC_B SYNC_A
        WSTAMP(1);
        ISSUE(NOSYNC, 0) ISSUE(NOSYNC, 1) ISSUE(NOSYNC, 2) ISSUE(NOSYNC, 3)
        WSTAMP(2);
        lb ^= (lds0 + w * 3072) ^ (lds0 + w * 3072 + 4 * BLKB);
        const u32 lbx = (lds0 + w * 3072) ^ (lds0 + w * 3072 + 4 * BLKB);
        ISSUE(SYNC_A, 0) ISSUE(SYNC_B, 1) ISSUE(SYNC_A, 2) ISSUE(SYNC_B, 3)
#pragma unroll 1
        for (int s = 0; s < 6; ++s) {
            sox = (sox + adj) & 0xf80u; adj ^= adjv; lb ^= lbx;
            ISSUE(SYNC_A, 0) ISSUE(SYNC_B, 1) ISSUE(SYNC_A, 2) ISSUE(SYNC_B, 3)
        }
        WSTAMP(3);
        if (PB >= 6) { wait_vm<15>(); BARRIER(); }
        if (PB >= 5) { wait_vm<12>(); BARRIER(); }
        wait_vm<9>(); BARRIER();
        wait_vm<6>(); BARRIER();
        wait_vm<3>(); BARRIER();
        wait_vm<0>(); BARRIER();
        WSTAMP(4);
        if (w != 0) { WDUMP(); return; }
        const int rank = p.rank;
        GAS char* myws = (GAS char*)p.myws;
        GAS u32* fl = (GAS u32*)((GAS char*)p.ws[dst] + WS_FLAGS) + rank * 32 + i;
        u32 call = __hip_atomic_load((GAS u32*)(myws + WS_CNT) + b, __ATOMIC_RELAXED, __HIP_MEMORY_SCOPE_AGENT) + 1;
        call = __builtin_amdgcn_readfirstlane(call);
        if (lane == 0) __hip_atomic_store((GAS u32*)(myws + WS_CNT) + b, call, __ATOMIC_RELAXED, __HIP_MEMORY_SCOPE_AGENT);
        asm volatile("" : "+s"(fl));
        BARRIER();
        WSTAMP(5);
        if (dst != rank) {
            if (lane == 0) __hip_atomic_store(fl, call, __ATOMIC_RELAXED, __HIP_MEMORY_SCOPE_SYSTEM);
        } else if (i == 0) {
            GAS const char* fp = myws + WS_FLAGS + lane * 16;
            const bool skip = (lane >> 3) == rank;
            const unsigned long long t0 = __builtin_amdgcn_s_memrealtime();
            for (;;) {
                u32x4 f;
                asm volatile("global_load_dwordx4 %0, %1, off sc0 sc1\n\ts_waitcnt vmcnt(0)" : "=&v"(f) : "v"(fp) : "memory");
                bool ok = skip || (f[0] >= call && f[1] >= call && f[2] >= call && f[3] >= call);
                if (__all(ok)) break;
                if (__builtin_amdgcn_s_memrealtime() - t0 > DEADLINE_TICKS) {
                    if (lane == 0) {
                        __hip_atomic_store((GAS u32*)(myws + WS_TMO), call, __ATOMIC_RELAXED, __HIP_MEMORY_SCOPE_AGENT);
                        ((GAS unsigned short*)p.out[dst])[(size_t)rank * MLOC * NLOC] = 0x7FC0;
                    }
                    break;
                }
                __builtin_amdgcn_s_sleep(1);
            }
        }
        WSTAMP(6);
        WDUMP();
        return;
    }

    const int c = w - 4, mi = c & 1, nh = c >> 1;
    const char* px = (const char*)(((unsigned long long)kxw[1] << 32) | kxw[0]);
    const bool c3 = C3FIX && (g & 1) == 0;
    if (lead && !(c3 && nb == 3)) {
        for (int k = 0; k < c; ++k) __builtin_amdgcn_s_sleep(TDELAY);
        const int bb = (jc + 2 + ((2 * c + (lane >> 5)) << 2) + rot) & 31;
        GLDS4(px, (u32)((m0 + (lane & 31)) * (KD * 2) + bb * 128), lds0 + RINGB + c * 256, "");
        if (c3 && nb == 1) {
            const int b2 = (((jc + 1) & 3) + 2 + ((2 * c + (lane >> 5)) << 2) + rot) & 31;
            GLDS4(px, (u32)((m0 + (lane & 31)) * (KD * 2) + b2 * 128), lds0 + RINGB + c * 256, "");
        }
    }
    GAS char* op = (GAS char*)p.out[dst] + ((size_t)(p.rank * MLOC + m0 + mi * 16 + (lane & 15)) * NLOC + nl0 + nh * 32 + ((lane >> 4) & 1) * 16 + (lane >> 5) * 8) * 2;
    const int c16 = lane & 15, q = lane >> 4;
    u32 xa[2], wa[2];
#pragma unroll
    for (int s = 0; s < 2; ++s) xa[s] = lds0 + (2 * mi + (c16 >> 3)) * 3072 + (c16 & 7) * 128 + (((4 * s + q) ^ (c16 >> 1)) * 16);
    {
        const int qq = c16 >> 2, pp = c16 & 3, x = ((qq >> 1) & 1) | ((q & 1) << 1);
#pragma unroll
        for (int t = 0; t < 2; ++t) wa[t] = lds0 + (q >> 1) * 3072 + 1024 + (q & 1) * 1024 + qq * 128 + (((2 * nh + t) ^ x) * 32) + pp * 8;
    }
    asm volatile("s_waitcnt lgkmcnt(0)" : "+v"(op), "+v"(xa[0]), "+v"(xa[1]), "+v"(wa[0]), "+v"(wa[1]));
    f32x4 acc[2] = {{0.f, 0.f, 0.f, 0.f}, {0.f, 0.f, 0.f, 0.f}};
    WSTAMP(1);
#define RDSTEP(u)                                                                                          \
    asm volatile("ds_read_b128 %0, %5 offset:%8\n\t"                                                       \
                 "ds_read_b64_tr_b16 %1, %6 offset:%9\n\tds_read_b64_tr_b16 %2, %6 offset:%10\n\t"         \
                 "ds_read_b64_tr_b16 %3, %7 offset:%9\n\tds_read_b64_tr_b16 %4, %7 offset:%10"             \
                 : "=&v"(xf[(u) & 3]), "=&v"(wf[(u) & 3][0]), "=&v"(wf[(u) & 3][1]), "=&v"(wf[(u) & 3][2]), "=&v"(wf[(u) & 3][3]) \
                 : "v"(xa[(u) & 1]), "v"(wa[0]), "v"(wa[1]), "i"(((u) >> 1) * BLKB),                       \
                   "i"(((u) >> 1) * BLKB + ((u) & 1) * 6144), "i"(((u) >> 1) * BLKB + ((u) & 1) * 6144 + 512) : "memory")
#define MMASTEP(u, N)                                                                                      \
    do {                                                                                                   \
        asm volatile("s_waitcnt lgkmcnt(" #N ")" : "+v"(xf[u]), "+v"(wf[u][0]), "+v"(wf[u][1]), "+v"(wf[u][2]), "+v"(wf[u][3]) :: "memory"); \
        u32x4 t0 = {wf[u][0][0], wf[u][0][1], wf[u][1][0], wf[u][1][1]};                                   \
        u32x4 t1 = {wf[u][2][0], wf[u][2][1], wf[u][3][0], wf[u][3][1]};                                   \
        acc[0] = __builtin_amdgcn_mfma_f32_16x16x32_bf16(__builtin_bit_cast(bf16x8, t0), __builtin_bit_cast(bf16x8, xf[u]), acc[0], 0, 0, 0); \
        acc[1] = __builtin_amdgcn_mfma_f32_16x16x32_bf16(__builtin_bit_cast(bf16x8, t1), __builtin_bit_cast(bf16x8, xf[u]), acc[1], 0, 0, 0); \
        __builtin_amdgcn_sched_barrier(0);                                                                 \
    } while (0)
#define BAR_B() BARRIER()
    u32x4 xf[4];
    u32x2 wf[4][4];
#pragma unroll
    for (int u = 2; u < 4; ++u) {
        xf[u] = (u32x4){0u, 0u, 0u, 0u};
#pragma unroll
        for (int k = 0; k < 4; ++k) wf[u][k] = (u32x2){0u, 0u};
    }
    int delta = 4 * BLKB;
    {
#pragma unroll 1
    for (int st = 0; st < 8; ++st) {
        BARRIER();
        RDSTEP(0); MMASTEP(2, 10);
        RDSTEP(1); MMASTEP(3, 10);
        BAR_B();
        RDSTEP(2); MMASTEP(0, 10);
        RDSTEP(3); MMASTEP(1, 10);
        BARRIER();
        RDSTEP(4); MMASTEP(2, 10);
        RDSTEP(5); MMASTEP(3, 10);
        BAR_B();
        RDSTEP(6); MMASTEP(0, 10);
        RDSTEP(7);
        xa[0] += delta; xa[1] += delta; wa[0] += delta; wa[1] += delta;
        delta = -delta;
        MMASTEP(1, 10);
    }
    }
    MMASTEP(2, 5);
    MMASTEP(3, 0);
    WSTAMP(7);
    {
        const auto s0 = __builtin_amdgcn_permlane16_swap(pack_bf16(acc[0][0], acc[0][1]), pack_bf16(acc[1][0], acc[1][1]), false, false);
        const auto s1 = __builtin_amdgcn_permlane16_swap(pack_bf16(acc[0][2], acc[0][3]), pack_bf16(acc[1][2], acc[1][3]), false, false);
        const u32x4 o = {s0[0], s1[0], s0[1], s1[1]};
        asm volatile("global_store_dwordx4 %0, %1, off sc0 sc1\n\ts_nop 1" :: "v"(op), "v"(o) : "memory");
    }
    WSTAMP(8);
    wait_vm<0>();
    WSTAMP(9);
    BARRIER();
    WDUMP();
}

extern "C" void kernel_launch(int rank, int nranks, ncclComm_t comm,
                              hipStream_t stream,
                              void* const* d_in, const int* in_sizes, int n_in,
                              void* d_out, int out_size,
                              const int* in_dtypes, int out_dtype,
                              void* const* d_out_all,
                              void* const* d_ws_all, size_t ws_bytes) {
    if (nranks != NR || (unsigned)rank >= (unsigned)NR || n_in < 2 || ws_bytes < WS_NEED ||
        in_sizes[0] != MLOC * KD || in_sizes[1] != KD * ND || out_size != NR * MLOC * NLOC) {
        fprintf(stderr, "kernel_launch: unexpected problem shape\n");
        return;
    }
    Params p;
    const unsigned long long ax = (unsigned long long)d_in[0], aw = (unsigned long long)d_in[1] - 1024;
    if ((ax >> 48) || (aw >> 48)) { fprintf(stderr, "kernel_launch: address beyond 48 bits\n"); return; }
    p.rx[0] = (u32)ax; p.rx[1] = (u32)(ax >> 32); p.rx[2] = MLOC * KD * 2; p.rx[3] = 0x00020000u;
    p.rw[0] = (u32)aw; p.rw[1] = (u32)(aw >> 32); p.rw[2] = KD * ND * 2 + 1024; p.rw[3] = 0x00020000u;
    for (int r = 0; r < NR; ++r) { p.out[r] = d_out_all[r]; p.ws[r] = d_ws_all[r]; }
    p.rank = rank; p.pad = 0;
    p.myws = d_ws_all[rank];
    hipLaunchKernelGGL(gemm_a2a, dim3(NWG), dim3(NTHR), 0, stream, p);
    HIP_CHECK(hipGetLastError());
}
```

```cpp
#include <hip/hip_runtime.h>
#include <rccl/rccl.h>
#include <cstdio>

#define HIP_CHECK(expr) do { hipError_t _e = (expr); if (_e != hipSuccess) { \
    fprintf(stderr, "HIP error %s at %s:%d\n", hipGetErrorString(_e), __FILE__, __LINE__); \
    return; } } while (0)

#define BPB 1
#define XTRA 0
#define LEAD 1
#define TDELAY 7
#define ROT (4 * g)
#define C3FIX 1

typedef unsigned int u32;
typedef u32 u32x2 __attribute__((ext_vector_type(2)));
typedef u32 u32x4 __attribute__((ext_vector_type(4)));
typedef u32 u32x8 __attribute__((ext_vector_type(8)));
typedef float f32x4 __attribute__((ext_vector_type(4)));
typedef __bf16 bf16x8 __attribute__((ext_vector_type(8)));
typedef __bf16 bf16x2 __attribute__((ext_vector_type(2)));
typedef float f32x2 __attribute__((ext_vector_type(2)));

constexpr int NR = 8;
constexpr int MLOC = 256;
constexpr int KD = 2048;
constexpr int ND = 2048;
constexpr int NLOC = 256;
constexpr int NWG = 256;
constexpr int NTHR = 512;
constexpr int BLKB = 12288;
constexpr int RINGB = 8 * BLKB;
constexpr int PB = 4 + XTRA;
constexpr size_t WS_FLAGS = 0;
constexpr size_t WS_CNT = 4096;
constexpr size_t WS_TMO = 8192;
constexpr size_t WS_NEED = 16384;
constexpr unsigned long long DEADLINE_TICKS = 50000000ull;

#define GAS __attribute__((address_space(1)))
#define LAS __attribute__((address_space(3)))
struct Params {
    u32 rx[4];
    u32 rw[4];
    void* myws;
    int rank, pad;
    void* out[NR];
    void* ws[NR];
};
#define WSTAMP(k)
#define WDUMP()

__device__ __forceinline__ u32 pack_bf16(float a, float b) {
    f32x2 v = {a, b};
    bf16x2 r = __builtin_convertvector(v, bf16x2);
    return __builtin_bit_cast(u32, r);
}
#define GLDS4(sbase, voff, lds_dst, bits) \
    asm volatile("s_mov_b32 m0, %2\n\ts_nop 0\n\tglobal_load_lds_dword %0, %1 " bits :: "v"(voff), "s"(sbase), "s"(lds_dst) : "memory")
#define BARRIER() do { asm volatile("" ::: "memory"); __builtin_amdgcn_s_barrier(); asm volatile("" ::: "memory"); } while (0)
template <int N> __device__ __forceinline__ void wait_vm() { asm volatile("s_waitcnt vmcnt(%0)" :: "n"(N) : "memory"); }

__global__ void __launch_bounds__(NTHR, 2) gemm_a2a(Params p) {
    __shared__ __attribute__((aligned(1024))) char smem[RINGB + 1024];
    u32x8 kxw;
    asm volatile("s_load_dwordx8 %0, %1, 0x0" : "=s"(kxw) : "s"(__builtin_amdgcn_kernarg_segment_ptr()));
    __builtin_amdgcn_sched_barrier(0);
    const int b = blockIdx.x;
    const int g = b & 7;
    const int i = b >> 3;
    const int mb = i & 7, nb = i >> 3;
    const int dst = (g + 2 * nb) & 7;
    const int m0 = mb * 32;
    const int nl0 = nb * 64;
    const int tid = threadIdx.x;
    const int lane = tid & 63;
    const int w = __builtin_amdgcn_readfirstlane(tid >> 6);
    WSTAMP(0);
    const u32 lds0 = (u32)(size_t)smem;
    const bool lead = LEAD && (((mb + nb) & 1) == 0);
    const int jc = ((mb >> 1) + (nb >> 1)) & 3;
    const int rot = ROT;

    const int r8 = lane >> 3, c8 = lane & 7;
    const u32 vx = (u32)((m0 + 8 * w + r8) * (KD * 2) + ((c8 ^ ((w & 1) * 4 + (r8 >> 1))) * 16));
    const u32 vwa = (u32)((16 * w + r8) * (ND * 2) + (dst * NLOC + nl0) * 2 + ((c8 ^ (r8 & 2)) * 16));
    const u32 vwb = (vwa ^ 64u) + (8 * ND * 2 - 1024);
    u32 sox = (u32)((((lead ? jc : 0) + rot) & 31) * 128);
    const u32 step = lead ? 512u : 128u, adjv = lead ? 128u : 0u;
    u32 lb = lds0 + w * 3072;
    asm volatile("s_waitcnt lgkmcnt(0)" : "+s"(kxw) : "s"(sox), "s"(lb), "v"(vx), "v"(vwa), "v"(vwb), "s"(step));
    const u32x4 rx = {kxw[0], kxw[1], kxw[2], kxw[3]};
    const u32x4 rw = {kxw[4], kxw[5], kxw[6], kxw[7]};

    if (__builtin_expect(w < 4, 1)) {
        u32 adj = adjv;
#define ISSUE(pre, slot)                                                                           \
        { u32 sow;                                                                                 \
        asm volatile("s_lshl_b32 %1, %0, 11\n\t"                                                   \
                     "s_add_u32 m0, %7, %9\n\t"                                                    \
                     pre                                                                           \
                     "buffer_load_dwordx4 %2, %5, %0 offen lds\n\t"                                \
                     "buffer_load_dwordx4 %3, %6, %1 offen offset:1024 lds\n\t"                    \
                     "buffer_load_dwordx4 %4, %6, %1 offen offset:2048 lds\n\t"                    \
                     "s_add_u32 %0, %0, %8\n\t"                                                    \
                     "s_and_b32 %0, %0, 0xf80"                                                     \
                     : "+s"(sox), "=&s"(sow) : "v"(vx), "v"(vwa), "v"(vwb), "s"(rx), "s"(rw), "s"(lb), "s"(step), "i"((slot) * BLKB) : "memory", "scc"); }
#define STR2(x) #x
#define STR(x) STR2(x)
#define NOSYNC "s_nop 0\n\t"
#define SYNC_A "s_waitcnt vmcnt(9)\n\ts_barrier\n\t"
#define SYNC_B SYNC_A
        WSTAMP(1);
        ISSUE(NOSYNC, 0) ISSUE(NOSYNC, 1) ISSUE(NOSYNC, 2) ISSUE(NOSYNC, 3)
        WSTAMP(2);
        lb ^= (lds0 + w * 3072) ^ (lds0 + w * 3072 + 4 * BLKB);
        const u32 lbx = (lds0 + w * 3072) ^ (lds0 + w * 3072 + 4 * BLKB);
        ISSUE(SYNC_A, 0) ISSUE(SYNC_B, 1) ISSUE(SYNC_A, 2) ISSUE(SYNC_B, 3)
#pragma unroll 1
        for (int s = 0; s < 6; ++s) {
            sox = (sox + adj) & 0xf80u; adj ^= adjv; lb ^= lbx;
            ISSUE(SYNC_A, 0) ISSUE(SYNC_B, 1) ISSUE(SYNC_A, 2) ISSUE(SYNC_B, 3)
        }
        WSTAMP(3);
        if (PB >= 6) { wait_vm<15>(); BARRIER(); }
        if (PB >= 5) { wait_vm<12>(); BARRIER(); }
        wait_vm<9>(); BARRIER();
        wait_vm<6>(); BARRIER();
        wait_vm<3>(); BARRIER();
        wait_vm<0>(); BARRIER();
        WSTAMP(4);
        if (w != 0) { WDUMP(); return; }
        const int rank = p.rank;
        GAS char* myws = (GAS char*)p.myws;
        GAS u32* fl = (GAS u32*)((GAS char*)p.ws[dst] + WS_FLAGS) + rank * 32 + i;
        u32 call = __hip_atomic_load((GAS u32*)(myws + WS_CNT) + b, __ATOMIC_RELAXED, __HIP_MEMORY_SCOPE_AGENT) + 1;
        call = __builtin_amdgcn_readfirstlane(call);
        if (lane == 0) __hip_atomic_store((GAS u32*)(myws + WS_CNT) + b, call, __ATOMIC_RELAXED, __HIP_MEMORY_SCOPE_AGENT);
        asm volatile("" : "+s"(fl));
        BARRIER();
        WSTAMP(5);
        if (dst != rank) {
            if (lane == 0) __hip_atomic_store(fl, call, __ATOMIC_RELAXED, __HIP_MEMORY_SCOPE_SYSTEM);
        } else if (i == 0) {
            GAS const char* fp = myws + WS_FLAGS + lane * 16;
            const bool skip = (lane >> 3) == rank;
            const unsigned long long t0 = __builtin_amdgcn_s_memrealtime();
            for (;;) {
                u32x4 f;
                asm volatile("global_load_dwordx4 %0, %1, off sc0 sc1\n\ts_waitcnt vmcnt(0)" : "=&v"(f) : "v"(fp) : "memory");
                bool ok = skip || (f[0] >= call && f[1] >= call && f[2] >= call && f[3] >= call);
                if (__all(ok)) break;
                if (__builtin_amdgcn_s_memrealtime() - t0 > DEADLINE_TICKS) {
                    if (lane == 0) {
                        __hip_atomic_store((GAS u32*)(myws + WS_TMO), call, __ATOMIC_RELAXED, __HIP_MEMORY_SCOPE_AGENT);
                        ((GAS unsigned short*)p.out[dst])[(size_t)rank * MLOC * NLOC] = 0x7FC0;
                    }
                    break;
                }
                __builtin_amdgcn_s_sleep(1);
            }
        }
        WSTAMP(6);
        WDUMP();
        return;
    }

    const int c = w - 4, mi = c & 1, nh = c >> 1;
    const char* px = (const char*)(((unsigned long long)kxw[1] << 32) | kxw[0]);
    const bool c3 = C3FIX && ((g ^ (kxw[4] >> 9)) & 1) == 0;
    if (lead && !(c3 && nb == 3)) {
        for (int k = 0; k < c; ++k) __builtin_amdgcn_s_sleep(TDELAY);
        const int bb = (jc + 2 + ((2 * c + (lane >> 5)) << 2) + rot) & 31;
        GLDS4(px, (u32)((m0 + (lane & 31)) * (KD * 2) + bb * 128), lds0 + RINGB + c * 256, "");
        if (c3 && nb == 1) {
            const int b2 = (((jc + 1) & 3) + 2 + ((2 * c + (lane >> 5)) << 2) + rot) & 31;
            GLDS4(px, (u32)((m0 + (lane & 31)) * (KD * 2) + b2 * 128), lds0 + RINGB + c * 256, "");
        }
    }
    GAS char* op = (GAS char*)p.out[dst] + ((size_t)(p.rank * MLOC + m0 + mi * 16 + (lane & 15)) * NLOC + nl0 + nh * 32 + ((lane >> 4) & 1) * 16 + (lane >> 5) * 8) * 2;
    const int c16 = lane & 15, q = lane >> 4;
    u32 xa[2], wa[2];
#pragma unroll
    for (int s = 0; s < 2; ++s) xa[s] = lds0 + (2 * mi + (c16 >> 3)) * 3072 + (c16 & 7) * 128 + (((4 * s + q) ^ (c16 >> 1)) * 16);
    {
        const int qq = c16 >> 2, pp = c16 & 3, x = ((qq >> 1) & 1) | ((q & 1) << 1);
#pragma unroll
        for (int t = 0; t < 2; ++t) wa[t] = lds0 + (q >> 1) * 3072 + 1024 + (q & 1) * 1024 + qq * 128 + (((2 * nh + t) ^ x) * 32) + pp * 8;
    }
    asm volatile("s_waitcnt lgkmcnt(0)" : "+v"(op), "+v"(xa[0]), "+v"(xa[1]), "+v"(wa[0]), "+v"(wa[1]));
    f32x4 acc[2] = {{0.f, 0.f, 0.f, 0.f}, {0.f, 0.f, 0.f, 0.f}};
    WSTAMP(1);
    u32 cnt, dl;
    asm volatile(
        "s_mov_b32 %[cnt], 8\n\ts_mov_b32 %[dl], 49152\n\t"
        "s_barrier\n\t"
        "ds_read_b128 v[64:67], %[xa0] offset:0\n\t"
        "ds_read_b64_tr_b16 v[68:69], %[wa0] offset:0\n\tds_read_b64_tr_b16 v[70:71], %[wa0] offset:512\n\t"
        "ds_read_b64_tr_b16 v[72:73], %[wa1] offset:0\n\tds_read_b64_tr_b16 v[74:75], %[wa1] offset:512\n\t"
        "ds_read_b128 v[76:79], %[xa1] offset:0\n\t"
        "ds_read_b64_tr_b16 v[80:81], %[wa0] offset:6144\n\tds_read_b64_tr_b16 v[82:83], %[wa0] offset:6656\n\t"
        "ds_read_b64_tr_b16 v[84:85], %[wa1] offset:6144\n\tds_read_b64_tr_b16 v[86:87], %[wa1] offset:6656\n\t"
        "1:\n\t"
        "s_barrier\n\t"
        "ds_read_b128 v[88:91], %[xa0] offset:12288\n\t"
        "ds_read_b64_tr_b16 v[92:93], %[wa0] offset:12288\n\tds_read_b64_tr_b16 v[94:95], %[wa0] offset:12800\n\t"
        "ds_read_b64_tr_b16 v[96:97], %[wa1] offset:12288\n\tds_read_b64_tr_b16 v[98:99], %[wa1] offset:12800\n\t"
        "s_waitcnt lgkmcnt(10)\n\t"
        "v_mfma_f32_16x16x32_bf16 %[a0], v[68:71], v[64:67], %[a0]\n\tv_mfma_f32_16x16x32_bf16 %[a1], v[72:75], v[64:67], %[a1]\n\t"
        "ds_read_b128 v[100:103], %[xa1] offset:12288\n\t"
        "ds_read_b64_tr_b16 v[104:105], %[wa0] offset:18432\n\tds_read_b64_tr_b16 v[106:107], %[wa0] offset:18944\n\t"
        "ds_read_b64_tr_b16 v[108:109], %[wa1] offset:18432\n\tds_read_b64_tr_b16 v[110:111], %[wa1] offset:18944\n\t"
        "s_waitcnt lgkmcnt(10)\n\t"
        "v_mfma_f32_16x16x32_bf16 %[a0], v[80:83], v[76:79], %[a0]\n\tv_mfma_f32_16x16x32_bf16 %[a1], v[84:87], v[76:79], %[a1]\n\t"
        "s_barrier\n\t"
        "ds_read_b128 v[64:67], %[xa0] offset:24576\n\t"
        "ds_read_b64_tr_b16 v[68:69], %[wa0] offset:24576\n\tds_read_b64_tr_b16 v[70:71], %[wa0] offset:25088\n\t"
        "ds_read_b64_tr_b16 v[72:73], %[wa1] offset:24576\n\tds_read_b64_tr_b16 v[74:75], %[wa1] offset:25088\n\t"
        "s_waitcnt lgkmcnt(10)\n\t"
        "v_mfma_f32_16x16x32_bf16 %[a0], v[92:95], v[88:91], %[a0]\n\tv_mfma_f32_16x16x32_bf16 %[a1], v[96:99], v[88:91], %[a1]\n\t"
        "ds_read_b128 v[76:79], %[xa1] offset:24576\n\t"
        "ds_read_b64_tr_b16 v[80:81], %[wa0] offset:30720\n\tds_read_b64_tr_b16 v[82:83], %[wa0] offset:31232\n\t"
        "ds_read_b64_tr_b16 v[84:85], %[wa1] offset:30720\n\tds_read_b64_tr_b16 v[86:87], %[wa1] offset:31232\n\t"
        "s_waitcnt lgkmcnt(10)\n\t"
        "v_mfma_f32_16x16x32_bf16 %[a0], v[104:107], v[100:103], %[a0]\n\tv_mfma_f32_16x16x32_bf16 %[a1], v[108:111], v[100:103], %[a1]\n\t"
        "s_barrier\n\t"
        "ds_read_b128 v[88:91], %[xa0] offset:36864\n\t"
        "ds_read_b64_tr_b16 v[92:93], %[wa0] offset:36864\n\tds_read_b64_tr_b16 v[94:95], %[wa0] offset:37376\n\t"
        "ds_read_b64_tr_b16 v[96:97], %[wa1] offset:36864\n\tds_read_b64_tr_b16 v[98:99], %[wa1] offset:37376\n\t"
        "s_waitcnt lgkmcnt(10)\n\t"
        "v_mfma_f32_16x16x32_bf16 %[a0], v[68:71], v[64:67], %[a0]\n\tv_mfma_f32_16x16x32_bf16 %[a1], v[72:75], v[64:67], %[a1]\n\t"
        "ds_read_b128 v[100:103], %[xa1] offset:36864\n\t"
        "ds_read_b64_tr_b16 v[104:105], %[wa0] offset:43008\n\tds_read_b64_tr_b16 v[106:107], %[wa0] offset:43520\n\t"
        "ds_read_b64_tr_b16 v[108:109], %[wa1] offset:43008\n\tds_read_b64_tr_b16 v[110:111], %[wa1] offset:43520\n\t"
        "v_add_u32 %[xa0], %[dl], %[xa0]\n\tv_add_u32 %[xa1], %[dl], %[xa1]\n\tv_add_u32 %[wa0], %[dl], %[wa0]\n\tv_add_u32 %[wa1], %[dl], %[wa1]\n\t"
        "s_sub_u32 %[dl], 0, %[dl]\n\ts_sub_u32 %[cnt], %[cnt], 1\n\ts_cmp_eq_u32 %[cnt], 0\n\t"
        "s_waitcnt lgkmcnt(10)\n\t"
        "v_mfma_f32_16x16x32_bf16 %[a0], v[80:83], v[76:79], %[a0]\n\tv_mfma_f32_16x16x32_bf16 %[a1], v[84:87], v[76:79], %[a1]\n\t"
        "s_cbranch_scc1 2f\n\t"
        "s_barrier\n\t"
        "ds_read_b128 v[64:67], %[xa0] offset:0\n\t"
        "ds_read_b64_tr_b16 v[68:69], %[wa0] offset:0\n\tds_read_b64_tr_b16 v[70:71], %[wa0] offset:512\n\t"
        "ds_read_b64_tr_b16 v[72:73], %[wa1] offset:0\n\tds_read_b64_tr_b16 v[74:75], %[wa1] offset:512\n\t"
        "s_waitcnt lgkmcnt(10)\n\t"
        "v_mfma_f32_16x16x32_bf16 %[a0], v[92:95], v[88:91], %[a0]\n\tv_mfma_f32_16x16x32_bf16 %[a1], v[96:99], v[88:91], %[a1]\n\t"
        "ds_read_b128 v[76:79], %[xa1] offset:0\n\t"
        "ds_read_b64_tr_b16 v[80:81], %[wa0] offset:6144\n\tds_read_b64_tr_b16 v[82:83], %[wa0] offset:6656\n\t"
        "ds_read_b64_tr_b16 v[84:85], %[wa1] offset:6144\n\tds_read_b64_tr_b16 v[86:87], %[wa1] offset:6656\n\t"
        "s_waitcnt lgkmcnt(10)\n\t"
        "v_mfma_f32_16x16x32_bf16 %[a0], v[104:107], v[100:103], %[a0]\n\tv_mfma_f32_16x16x32_bf16 %[a1], v[108:111], v[100:103], %[a1]\n\t"
        "s_branch 1b\n\t"
        "2:\n\t"
        "s_waitcnt lgkmcnt(5)\n\t"
        "v_mfma_f32_16x16x32_bf16 %[a0], v[92:95], v[88:91], %[a0]\n\tv_mfma_f32_16x16x32_bf16 %[a1], v[96:99], v[88:91], %[a1]\n\t"
        "s_waitcnt lgkmcnt(0)\n\t"
        "v_mfma_f32_16x16x32_bf16 %[a0], v[104:107], v[100:103], %[a0]\n\tv_mfma_f32_16x16x32_bf16 %[a1], v[108:111], v[100:103], %[a1]\n\t"
        "s_nop 7"
        : [a0] "+v"(acc[0]), [a1] "+v"(acc[1]), [xa0] "+v"(xa[0]), [xa1] "+v"(xa[1]), [wa0] "+v"(wa[0]), [wa1] "+v"(wa[1]),
          [cnt] "=&s"(cnt), [dl] "=&s"(dl)
        :
        : "memory", "scc", "v64", "v65", "v66", "v67", "v68", "v69", "v70", "v71", "v72", "v73", "v74", "v75", "v76", "v77", "v78", "v79", "v80", "v81", "v82", "v83", "v84", "v85", "v86", "v87", "v88", "v89", "v90", "v91", "v92", "v93", "v94", "v95", "v96", "v97", "v98", "v99", "v100", "v101", "v102", "v103", "v104", "v105", "v106", "v107", "v108", "v109", "v110", "v111");
    WSTAMP(7);
    {
        const auto s0 = __builtin_amdgcn_permlane16_swap(pack_bf16(acc[0][0], acc[0][1]), pack_bf16(acc[1][0], acc[1][1]), false, false);
        const auto s1 = __builtin_amdgcn_permlane16_swap(pack_bf16(acc[0][2], acc[0][3]), pack_bf16(acc[1][2], acc[1][3]), false, false);
        const u32x4 o = {s0[0], s1[0], s0[1], s1[1]};
        asm volatile("global_store_dwordx4 %0, %1, off sc0 sc1\n\ts_nop 1" :: "v"(op), "v"(o) : "memory");
    }
    WSTAMP(8);
    wait_vm<0>();
    WSTAMP(9);
    BARRIER();
    WDUMP();
}

extern "C" void kernel_launch(int rank, int nranks, ncclComm_t comm,
                              hipStream_t stream,
                              void* const* d_in, const int* in_sizes, int n_in,
                              void* d_out, int out_size,
                              const int* in_dtypes, int out_dtype,
                              void* const* d_out_all,
                              void* const* d_ws_all, size_t ws_bytes) {
    if (nranks != NR || (unsigned)rank >= (unsigned)NR || n_in < 2 || ws_bytes < WS_NEED ||
        in_sizes[0] != MLOC * KD || in_sizes[1] != KD * ND || out_size != NR * MLOC * NLOC) {
        fprintf(stderr, "kernel_launch: unexpected problem shape\n");
        return;
    }
    Params p;
    const unsigned long long ax = (unsigned long long)d_in[0], aw = (unsigned long long)d_in[1] - 1024;
    if ((ax >> 48) || (aw >> 48)) { fprintf(stderr, "kernel_launch: address beyond 48 bits\n"); return; }
    p.rx[0] = (u32)ax; p.rx[1] = (u32)(ax >> 32); p.rx[2] = MLOC * KD * 2; p.rx[3] = 0x00020000u;
    p.rw[0] = (u32)aw; p.rw[1] = (u32)(aw >> 32); p.rw[2] = KD * ND * 2 + 1024; p.rw[3] = 0x00020000u;
    for (int r = 0; r < NR; ++r) { p.out[r] = d_out_all[r]; p.ws[r] = d_ws_all[r]; }
    p.rank = rank; p.pad = 0;
    p.myws = d_ws_all[rank];
    hipLaunchKernelGGL(gemm_a2a, dim3(NWG), dim3(NTHR), 0, stream, p);
    HIP_CHECK(hipGetLastError());
}
```

```cpp
#include <hip/hip_runtime.h>
#include <rccl/rccl.h>
#include <cstdio>

#define HIP_CHECK(expr) do { hipError_t _e = (expr); if (_e != hipSuccess) { \
    fprintf(stderr, "HIP error %s at %s:%d\n", hipGetErrorString(_e), __FILE__, __LINE__); \
    return; } } while (0)

typedef unsigned int u32;
typedef u32 u32x4 __attribute__((ext_vector_type(4)));
typedef u32 u32x8 __attribute__((ext_vector_type(8)));
typedef float f32x4 __attribute__((ext_vector_type(4)));
typedef float f32x2 __attribute__((ext_vector_type(2)));
typedef __bf16 bf16x2 __attribute__((ext_vector_type(2)));

constexpr int NR = 8;
constexpr int MLOC = 256;
constexpr int KD = 2048;
constexpr int ND = 2048;
constexpr int NLOC = 256;
constexpr int NWG = 256;
constexpr int NTHR = 512;
constexpr int BLKB = 12288;
constexpr int RINGB = 8 * BLKB;
constexpr size_t WS_FLAGS = 0;
constexpr size_t WS_CNT = 4096;
constexpr size_t WS_TMO = 8192;
constexpr size_t WS_NEED = 16384;
constexpr unsigned long long DEADLINE_TICKS = 50000000ull;

#define GAS __attribute__((address_space(1)))
struct Params {
    u32 rx[4];
    u32 rw[4];
    void* myws;
    int rank, pad;
    void* out[NR];
    void* ws[NR];
};

__device__ __forceinline__ u32 pack_bf16(float a, float b) {
    f32x2 v = {a, b};
    bf16x2 r = __builtin_convertvector(v, bf16x2);
    return __builtin_bit_cast(u32, r);
}
#define BARRIER() do { asm volatile("" ::: "memory"); __builtin_amdgcn_s_barrier(); asm volatile("" ::: "memory"); } while (0)
template <int N> __device__ __forceinline__ void wait_vm() { asm volatile("s_waitcnt vmcnt(%0)" :: "n"(N) : "memory"); }

__global__ void __launch_bounds__(NTHR, 2) gemm_a2a(Params p) {
    __shared__ __attribute__((aligned(1024))) char smem[RINGB + 1024];
    u32x8 kxw;
    asm volatile("s_load_dwordx8 %0, %1, 0x0" : "=s"(kxw) : "s"(__builtin_amdgcn_kernarg_segment_ptr()));
    __builtin_amdgcn_sched_barrier(0);
    const int b = blockIdx.x;
    const int g = b & 7;
    const int i = b >> 3;
    const int mb = i & 7, nb = i >> 3;
    const int dst = (g + 2 * nb) & 7;
    const int m0 = mb * 32;
    const int nl0 = nb * 64;
    const int lane = threadIdx.x & 63;
    const int w = __builtin_amdgcn_readfirstlane(threadIdx.x >> 6);
    const u32 lds0 = (u32)(size_t)smem;
    const bool lead = ((mb + nb) & 1) == 0;
    const int jc = ((mb >> 1) + (nb >> 1)) & 3;

    const int r8 = lane >> 3, c8 = lane & 7;
    const u32 vx = (u32)((m0 + 8 * w + r8) * (KD * 2) + ((c8 ^ ((w & 1) * 4 + (r8 >> 1))) * 16));
    const u32 vwa = (u32)((16 * w + r8) * (ND * 2) + (dst * NLOC + nl0) * 2 + ((c8 ^ (r8 & 2)) * 16));
    const u32 vwb = (vwa ^ 64u) + (8 * ND * 2 - 1024);
    u32 sox = (u32)((((lead ? jc : 0) + 4 * g) & 31) * 128);
    const u32 step = lead ? 512u : 128u, adjv = lead ? 128u : 0u;
    u32 lb = lds0 + w * 3072;
    asm volatile("s_waitcnt lgkmcnt(0)" : "+s"(kxw) : "s"(sox), "s"(lb), "v"(vx), "v"(vwa), "v"(vwb), "s"(step));
    const u32x4 rx = {kxw[0], kxw[1], kxw[2], kxw[3]};
    const u32x4 rw = {kxw[4], kxw[5], kxw[6], kxw[7]};

    if (__builtin_expect(w < 4, 1)) {
#define ISSUE(pre, slot)                                                                           \
        { u32 sow;                                                                                 \
        asm volatile("s_lshl_b32 %1, %0, 11\n\t"                                                   \
                     "s_add_u32 m0, %7, %9\n\t"                                                    \
                     pre                                                                           \
                     "buffer_load_dwordx4 %2, %5, %0 offen lds\n\t"                                \
                     "buffer_load_dwordx4 %3, %6, %1 offen offset:1024 lds\n\t"                    \
                     "buffer_load_dwordx4 %4, %6, %1 offen offset:2048 lds\n\t"                    \
                     "s_add_u32 %0, %0, %8\n\t"                                                    \
                     "s_and_b32 %0, %0, 0xf80"                                                     \
                     : "+s"(sox), "=&s"(sow) : "v"(vx), "v"(vwa), "v"(vwb), "s"(rx), "s"(rw), "s"(lb), "s"(step), "i"((slot) * BLKB) : "memory", "scc"); }
#define TOUCH()                                                                                    \
        { u32 tmp;                                                                                 \
        asm volatile("s_mov_b32 m0, %3\n\t"                                                        \
                     "v_add_u32 %1, 0x200, %0\n\t"                                                 \
                     "buffer_load_dword %0, %2, 0 offen lds\n\t"                                   \
                     "v_bfi_b32 %0, %4, %1, %0"                                                    \
                     : "+v"(vt), "=&v"(tmp) : "s"(rt), "s"(sink), "s"(0xf80u) : "memory"); }
#define ISSUET(pre, slot) ISSUE(pre, slot) TOUCH()
#define NOSYNC "s_nop 0\n\t"
#define SYNC(n) "s_waitcnt vmcnt(" #n ")\n\ts_barrier\n\t"
        const u32 sox0 = sox;
        ISSUE(NOSYNC, 0)
        __builtin_amdgcn_sched_barrier(0);
        const bool c3 = ((g ^ (kxw[4] >> 9)) & 1) == 0;
        const u32x4 rt = {kxw[0], kxw[1], (lead && !(c3 && nb == 3)) ? kxw[2] : 0u, kxw[3]};
        u32 vt = (vx & ~0xfffu) | ((sox0 + 256u + ((c3 && nb == 1) ? (lane & 1) * 128u : 0u)) & 0xf80u);
        const u32 sink = lds0 + RINGB + w * 256;
        const u32 lbx = (lds0 + w * 3072) ^ (lds0 + w * 3072 + 4 * BLKB);
        u32 adj = adjv;
        TOUCH()
        ISSUET(NOSYNC, 1) ISSUET(NOSYNC, 2) ISSUET(NOSYNC, 3)
        lb ^= lbx;
        ISSUET(SYNC(13), 0) ISSUET(SYNC(13), 1) ISSUET(SYNC(13), 2) ISSUET(SYNC(13), 3)
        sox = (sox + adj) & 0xf80u; adj ^= adjv; lb ^= lbx;
        ISSUE(SYNC(13), 0) ISSUE(SYNC(12), 1) ISSUE(SYNC(11), 2) ISSUE(SYNC(10), 3)
#pragma unroll 1
        for (int s = 0; s < 5; ++s) {
            sox = (sox + adj) & 0xf80u; adj ^= adjv; lb ^= lbx;
            ISSUE(SYNC(9), 0) ISSUE(SYNC(9), 1) ISSUE(SYNC(9), 2) ISSUE(SYNC(9), 3)
        }
        wait_vm<9>(); BARRIER();
        wait_vm<6>(); BARRIER();
        wait_vm<3>(); BARRIER();
        wait_vm<0>(); BARRIER();
        if (w != 0) return;
        const int rank = p.rank;
        GAS char* myws = (GAS char*)p.myws;
        GAS u32* fl = (GAS u32*)((GAS char*)p.ws[dst] + WS_FLAGS) + rank * 32 + i;
        u32 call = __hip_atomic_load((GAS u32*)(myws + WS_CNT) + b, __ATOMIC_RELAXED, __HIP_MEMORY_SCOPE_AGENT) + 1;
        call = __builtin_amdgcn_readfirstlane(call);
        if (lane == 0) __hip_atomic_store((GAS u32*)(myws + WS_CNT) + b, call, __ATOMIC_RELAXED, __HIP_MEMORY_SCOPE_AGENT);
        asm volatile("" : "+s"(fl));
        BARRIER();
        if (dst != rank) {
            if (lane == 0) __hip_atomic_store(fl, call, __ATOMIC_RELAXED, __HIP_MEMORY_SCOPE_SYSTEM);
        } else if (i == 0) {
            GAS const char* fp = myws + WS_FLAGS + lane * 16;
            bool ok = (lane >> 3) == rank;
            const unsigned long long t0 = __builtin_amdgcn_s_memrealtime();
            for (;;) {
                const unsigned long long now = __builtin_amdgcn_s_memrealtime();
                if (!ok) {
                    u32x4 f;
                    asm volatile("global_load_dwordx4 %0, %1, off sc0 sc1\n\ts_waitcnt vmcnt(0)" : "=&v"(f) : "v"(fp) : "memory");
                    ok = f[0] >= call && f[1] >= call && f[2] >= call && f[3] >= call;
                }
                if (__all(ok)) break;
                if (now - t0 > DEADLINE_TICKS) {
                    if (lane == 0) {
                        __hip_atomic_store((GAS u32*)(myws + WS_TMO), call, __ATOMIC_RELAXED, __HIP_MEMORY_SCOPE_AGENT);
                        ((GAS unsigned short*)p.out[dst])[(size_t)rank * MLOC * NLOC] = 0x7FC0;
                    }
                    break;
                }
            }
        }
        return;
    }

    const int c = w - 4, mi = c & 1, nh = c >> 1;
    const int c16 = lane & 15, q = lane >> 4;
    GAS char* op = (GAS char*)p.out[dst] + ((size_t)(p.rank * MLOC + m0 + mi * 16 + c16) * NLOC + nl0 + nh * 32 + (q & 1) * 16 + (q >> 1) * 8) * 2;
    u32 xa[2], wa[2];
#pragma unroll
    for (int s = 0; s < 2; ++s) xa[s] = lds0 + (2 * mi + (c16 >> 3)) * 3072 + (c16 & 7) * 128 + (((4 * s + q) ^ (c16 >> 1)) * 16);
    {
        const int qq = c16 >> 2, pp = c16 & 3, x = ((qq >> 1) & 1) | ((q & 1) << 1);
#pragma unroll
        for (int t = 0; t < 2; ++t) wa[t] = lds0 + (q >> 1) * 3072 + 1024 + (q & 1) * 1024 + qq * 128 + (((2 * nh + t) ^ x) * 32) + pp * 8;
    }
    asm volatile("s_waitcnt lgkmcnt(0)" : "+v"(op), "+v"(xa[0]), "+v"(xa[1]), "+v"(wa[0]), "+v"(wa[1]));
    f32x4 acc[2] = {{0.f, 0.f, 0.f, 0.f}, {0.f, 0.f, 0.f, 0.f}};
    u32 cnt, dl;
    asm volatile(
        "s_mov_b32 %[cnt], 8\n\ts_mov_b32 %[dl], 49152\n\t"
        "s_barrier\n\t"
        "ds_read_b128 v[64:67], %[xa0] offset:0\n\t"
        "ds_read_b64_tr_b16 v[68:69], %[wa0] offset:0\n\tds_read_b64_tr_b16 v[70:71], %[wa0] offset:512\n\t"
        "ds_read_b64_tr_b16 v[72:73], %[wa1] offset:0\n\tds_read_b64_tr_b16 v[74:75], %[wa1] offset:512\n\t"
        "ds_read_b128 v[76:79], %[xa1] offset:0\n\t"
        "ds_read_b64_tr_b16 v[80:81], %[wa0] offset:6144\n\tds_read_b64_tr_b16 v[82:83], %[wa0] offset:6656\n\t"
        "ds_read_b64_tr_b16 v[84:85], %[wa1] offset:6144\n\tds_read_b64_tr_b16 v[86:87], %[wa1] offset:6656\n\t"
        "1:\n\t"
        "s_barrier\n\t"
        "ds_read_b128 v[88:91], %[xa0] offset:12288\n\t"
        "ds_read_b64_tr_b16 v[92:93], %[wa0] offset:12288\n\tds_read_b64_tr_b16 v[94:95], %[wa0] offset:12800\n\t"
        "ds_read_b64_tr_b16 v[96:97], %[wa1] offset:12288\n\tds_read_b64_tr_b16 v[98:99], %[wa1] offset:12800\n\t"
        "s_waitcnt lgkmcnt(10)\n\t"
        "v_mfma_f32_16x16x32_bf16 %[a0], v[68:71], v[64:67], %[a0]\n\tv_mfma_f32_16x16x32_bf16 %[a1], v[72:75], v[64:67], %[a1]\n\t"
        "ds_read_b128 v[100:103], %[xa1] offset:12288\n\t"
        "ds_read_b64_tr_b16 v[104:105], %[wa0] offset:18432\n\tds_read_b64_tr_b16 v[106:107], %[wa0] offset:18944\n\t"
        "ds_read_b64_tr_b16 v[108:109], %[wa1] offset:18432\n\tds_read_b64_tr_b16 v[110:111], %[wa1] offset:18944\n\t"
        "s_waitcnt lgkmcnt(10)\n\t"
        "v_mfma_f32_16x16x32_bf16 %[a0], v[80:83], v[76:79], %[a0]\n\tv_mfma_f32_16x16x32_bf16 %[a1], v[84:87], v[76:79], %[a1]\n\t"
        "s_barrier\n\t"
        "ds_read_b128 v[64:67], %[xa0] offset:24576\n\t"
        "ds_read_b64_tr_b16 v[68:69], %[wa0] offset:24576\n\tds_read_b64_tr_b16 v[70:71], %[wa0] offset:25088\n\t"
        "ds_read_b64_tr_b16 v[72:73], %[wa1] offset:24576\n\tds_read_b64_tr_b16 v[74:75], %[wa1] offset:25088\n\t"
        "s_waitcnt lgkmcnt(10)\n\t"
        "v_mfma_f32_16x16x32_bf16 %[a0], v[92:95], v[88:91], %[a0]\n\tv_mfma_f32_16x16x32_bf16 %[a1], v[96:99], v[88:91], %[a1]\n\t"
        "ds_read_b128 v[76:79], %[xa1] offset:24576\n\t"
        "ds_read_b64_tr_b16 v[80:81], %[wa0] offset:30720\n\tds_read_b64_tr_b16 v[82:83], %[wa0] offset:31232\n\t"
        "ds_read_b64_tr_b16 v[84:85], %[wa1] offset:30720\n\tds_read_b64_tr_b16 v[86:87], %[wa1] offset:31232\n\t"
        "s_waitcnt lgkmcnt(10)\n\t"
        "v_mfma_f32_16x16x32_bf16 %[a0], v[104:107], v[100:103], %[a0]\n\tv_mfma_f32_16x16x32_bf16 %[a1], v[108:111], v[100:103], %[a1]\n\t"
        "s_barrier\n\t"
        "ds_read_b128 v[88:91], %[xa0] offset:36864\n\t"
        "ds_read_b64_tr_b16 v[92:93], %[wa0] offset:36864\n\tds_read_b64_tr_b16 v[94:95], %[wa0] offset:37376\n\t"
        "ds_read_b64_tr_b16 v[96:97], %[wa1] offset:36864\n\tds_read_b64_tr_b16 v[98:99], %[wa1] offset:37376\n\t"
        "s_waitcnt lgkmcnt(10)\n\t"
        "v_mfma_f32_16x16x32_bf16 %[a0], v[68:71], v[64:67], %[a0]\n\tv_mfma_f32_16x16x32_bf16 %[a1], v[72:75], v[64:67], %[a1]\n\t"
        "ds_read_b128 v[100:103], %[xa1] offset:36864\n\t"
        "ds_read_b64_tr_b16 v[104:105], %[wa0] offset:43008\n\tds_read_b64_tr_b16 v[106:107], %[wa0] offset:43520\n\t"
        "ds_read_b64_tr_b16 v[108:109], %[wa1] offset:43008\n\tds_read_b64_tr_b16 v[110:111], %[wa1] offset:43520\n\t"
        "v_add_u32 %[xa0], %[dl], %[xa0]\n\tv_add_u32 %[xa1], %[dl], %[xa1]\n\tv_add_u32 %[wa0], %[dl], %[wa0]\n\tv_add_u32 %[wa1], %[dl], %[wa1]\n\t"
        "s_sub_u32 %[dl], 0, %[dl]\n\ts_sub_u32 %[cnt], %[cnt], 1\n\ts_cmp_eq_u32 %[cnt], 0\n\t"
        "s_waitcnt lgkmcnt(10)\n\t"
        "v_mfma_f32_16x16x32_bf16 %[a0], v[80:83], v[76:79], %[a0]\n\tv_mfma_f32_16x16x32_bf16 %[a1], v[84:87], v[76:79], %[a1]\n\t"
        "s_cbranch_scc1 2f\n\t"
        "s_barrier\n\t"
        "ds_read_b128 v[64:67], %[xa0] offset:0\n\t"
        "ds_read_b64_tr_b16 v[68:69], %[wa0] offset:0\n\tds_read_b64_tr_b16 v[70:71], %[wa0] offset:512\n\t"
        "ds_read_b64_tr_b16 v[72:73], %[wa1] offset:0\n\tds_read_b64_tr_b16 v[74:75], %[wa1] offset:512\n\t"
        "s_waitcnt lgkmcnt(10)\n\t"
        "v_mfma_f32_16x16x32_bf16 %[a0], v[92:95], v[88:91], %[a0]\n\tv_mfma_f32_16x16x32_bf16 %[a1], v[96:99], v[88:91], %[a1]\n\t"
        "ds_read_b128 v[76:79], %[xa1] offset:0\n\t"
        "ds_read_b64_tr_b16 v[80:81], %[wa0] offset:6144\n\tds_read_b64_tr_b16 v[82:83], %[wa0] offset:6656\n\t"
        "ds_read_b64_tr_b16 v[84:85], %[wa1] offset:6144\n\tds_read_b64_tr_b16 v[86:87], %[wa1] offset:6656\n\t"
        "s_waitcnt lgkmcnt(10)\n\t"
        "v_mfma_f32_16x16x32_bf16 %[a0], v[104:107], v[100:103], %[a0]\n\tv_mfma_f32_16x16x32_bf16 %[a1], v[108:111], v[100:103], %[a1]\n\t"
        "s_branch 1b\n\t"
        "2:\n\t"
        "s_waitcnt lgkmcnt(5)\n\t"
        "v_mfma_f32_16x16x32_bf16 %[a0], v[92:95], v[88:91], %[a0]\n\tv_mfma_f32_16x16x32_bf16 %[a1], v[96:99], v[88:91], %[a1]\n\t"
        "s_waitcnt lgkmcnt(0)\n\t"
        "v_mfma_f32_16x16x32_bf16 %[a0], v[104:107], v[100:103], %[a0]\n\tv_mfma_f32_16x16x32_bf16 %[a1], v[108:111], v[100:103], %[a1]\n\t"
        "s_nop 7"
        : [a0] "+v"(acc[0]), [a1] "+v"(acc[1]), [xa0] "+v"(xa[0]), [xa1] "+v"(xa[1]), [wa0] "+v"(wa[0]), [wa1] "+v"(wa[1]),
          [cnt] "=&s"(cnt), [dl] "=&s"(dl)
        :
        : "memory", "scc", "v64", "v65", "v66", "v67", "v68", "v69", "v70", "v71", "v72", "v73", "v74", "v75", "v76", "v77", "v78", "v79", "v80", "v81", "v82", "v83", "v84", "v85", "v86", "v87", "v88", "v89", "v90", "v91", "v92", "v93", "v94", "v95", "v96", "v97", "v98", "v99", "v100", "v101", "v102", "v103", "v104", "v105", "v106", "v107", "v108", "v109", "v110", "v111");
    {
        const auto s0 = __builtin_amdgcn_permlane16_swap(pack_bf16(acc[0][0], acc[0][1]), pack_bf16(acc[1][0], acc[1][1]), false, false);
        const auto s1 = __builtin_amdgcn_permlane16_swap(pack_bf16(acc[0][2], acc[0][3]), pack_bf16(acc[1][2], acc[1][3]), false, false);
        const u32x4 o = {s0[0], s1[0], s0[1], s1[1]};
        asm volatile("global_store_dwordx4 %0, %1, off sc0 sc1\n\ts_nop 1" :: "v"(op), "v"(o) : "memory");
    }
    wait_vm<0>();
    BARRIER();
}

extern "C" void kernel_launch(int rank, int nranks, ncclComm_t comm,
                              hipStream_t stream,
                              void* const* d_in, const int* in_sizes, int n_in,
                              void* d_out, int out_size,
                              const int* in_dtypes, int out_dtype,
                              void* const* d_out_all,
                              void* const* d_ws_all, size_t ws_bytes) {
    if (nranks != NR || (unsigned)rank >= (unsigned)NR || n_in < 2 || ws_bytes < WS_NEED ||
        in_sizes[0] != MLOC * KD || in_sizes[1] != KD * ND || out_size != NR * MLOC * NLOC) {
        fprintf(stderr, "kernel_launch: unexpected problem shape\n");
        return;
    }
    Params p;
    const unsigned long long ax = (unsigned long long)d_in[0], aw = (unsigned long long)d_in[1] - 1024;
    if ((ax >> 48) || (aw >> 48)) { fprintf(stderr, "kernel_launch: address beyond 48 bits\n"); return; }
    p.rx[0] = (u32)ax; p.rx[1] = (u32)(ax >> 32); p.rx[2] = MLOC * KD * 2; p.rx[3] = 0x00020000u;
    p.rw[0] = (u32)aw; p.rw[1] = (u32)(aw >> 32); p.rw[2] = KD * ND * 2 + 1024; p.rw[3] = 0x00020000u;
    for (int r = 0; r < NR; ++r) { p.out[r] = d_out_all[r]; p.ws[r] = d_ws_all[r]; }
    p.rank = rank; p.pad = 0;
    p.myws = d_ws_all[rank];
    hipLaunchKernelGGL(gemm_a2a, dim3(NWG), dim3(NTHR), 0, stream, p);
    HIP_CHECK(hipGetLastError());
}
```

```cpp
#include <hip/hip_runtime.h>
#include <rccl/rccl.h>
#include <cstdio>

#define HIP_CHECK(expr) do { hipError_t _e = (expr); if (_e != hipSuccess) { \
    fprintf(stderr, "HIP error %s at %s:%d\n", hipGetErrorString(_e), __FILE__, __LINE__); \
    return; } } while (0)

typedef unsigned int u32;
typedef u32 u32x4 __attribute__((ext_vector_type(4)));
typedef u32 u32x8 __attribute__((ext_vector_type(8)));
typedef float f32x4 __attribute__((ext_vector_type(4)));
typedef float f32x2 __attribute__((ext_vector_type(2)));
typedef __bf16 bf16x2 __attribute__((ext_vector_type(2)));

constexpr int NR = 8;
constexpr int MLOC = 256;
constexpr int KD = 2048;
constexpr int ND = 2048;
constexpr int NLOC = 256;
constexpr int NWG = 256;
constexpr int NTHR = 512;
constexpr int BLKB = 12288;
constexpr int RINGB = 8 * BLKB;
constexpr size_t WS_FLAGS = 0;
constexpr size_t WS_CNT = 4096;
constexpr size_t WS_TMO = 8192;
constexpr size_t WS_NEED = 16384;
constexpr unsigned long long DEADLINE_TICKS = 50000000ull;

#define GAS __attribute__((address_space(1)))
struct Params {
    u32 rx[4];
    u32 rw[4];
    void* myws;
    int rank, pad;
    void* out[NR];
    void* ws[NR];
};

__device__ __forceinline__ u32 pack_bf16(float a, float b) {
    f32x2 v = {a, b};
    bf16x2 r = __builtin_convertvector(v, bf16x2);
    return __builtin_bit_cast(u32, r);
}
#define BARRIER() do { asm volatile("" ::: "memory"); __builtin_amdgcn_s_barrier(); asm volatile("" ::: "memory"); } while (0)
template <int N> __device__ __forceinline__ void wait_vm() { asm volatile("s_waitcnt vmcnt(%0)" :: "n"(N) : "memory"); }

__global__ void __launch_bounds__(NTHR, 2) gemm_a2a(Params p) {
    __shared__ __attribute__((aligned(1024))) char smem[RINGB + 1024];
    u32x8 kxw;
    asm volatile("s_load_dwordx8 %0, %1, 0x0" : "=s"(kxw) : "s"(__builtin_amdgcn_kernarg_segment_ptr()));
    __builtin_amdgcn_sched_barrier(0);
    const int b = blockIdx.x;
    const int g = b & 7;
    const int i = b >> 3;
    const int mb = i & 7, nb = i >> 3;
    const int dst = (g + 2 * nb) & 7;
    const int m0 = mb * 32;
    const int nl0 = nb * 64;
    const int lane = threadIdx.x & 63;
    const int w = __builtin_amdgcn_readfirstlane(threadIdx.x >> 6);
    const u32 lds0 = (u32)(size_t)smem;
    const bool lead = ((mb + nb) & 1) == 0;
    const int jc = ((mb >> 1) + (nb >> 1)) & 3;

    const int r8 = lane >> 3, c8 = lane & 7;
    const u32 vx = (u32)((m0 + 8 * w + r8) * (KD * 2) + ((c8 ^ ((w & 1) * 4 + (r8 >> 1))) * 16));
    const u32 vwa = (u32)((16 * w + r8) * (ND * 2) + (dst * NLOC + nl0) * 2 + ((c8 ^ (r8 & 2)) * 16));
    const u32 vwb = (vwa ^ 64u) + (8 * ND * 2 - 1024);
    u32 sox = (u32)((((lead ? jc : 0) + 4 * g) & 31) * 128);
    const u32 step = lead ? 512u : 128u, adjv = lead ? 128u : 0u;
    u32 lb = lds0 + w * 3072;
    asm volatile("s_waitcnt lgkmcnt(0)" : "+s"(kxw) : "s"(sox), "s"(lb), "v"(vx), "v"(vwa), "v"(vwb), "s"(step));
    const u32x4 rx = {kxw[0], kxw[1], kxw[2], kxw[3]};
    const u32x4 rw = {kxw[4], kxw[5], kxw[6], kxw[7]};

    if (__builtin_expect(w < 4, 1)) {
#define ISSUE(pre, slot)                                                                           \
        { u32 sow;                                                                                 \
        asm volatile("s_lshl_b32 %1, %0, 11\n\t"                                                   \
                     "s_add_u32 m0, %7, %9\n\t"                                                    \
                     pre                                                                           \
                     "buffer_load_dwordx4 %2, %5, %0 offen lds\n\t"                                \
                     "buffer_load_dwordx4 %3, %6, %1 offen offset:1024 lds\n\t"                    \
                     "buffer_load_dwordx4 %4, %6, %1 offen offset:2048 lds\n\t"                    \
                     "s_add_u32 %0, %0, %8\n\t"                                                    \
                     "s_and_b32 %0, %0, 0xf80"                                                     \
                     : "+s"(sox), "=&s"(sow) : "v"(vx), "v"(vwa), "v"(vwb), "s"(rx), "s"(rw), "s"(lb), "s"(step), "i"((slot) * BLKB) : "memory", "scc"); }
#define TOUCH()                                                                                    \
        { u32 tmp;                                                                                 \
        asm volatile("s_mov_b32 m0, %3\n\t"                                                        \
                     "v_add_u32 %1, 0x200, %0\n\t"                                                 \
                     "buffer_load_dword %0, %2, 0 offen lds\n\t"                                   \
                     "v_bfi_b32 %0, %4, %1, %0"                                                    \
                     : "+v"(vt), "=&v"(tmp) : "s"(rt), "s"(sink), "s"(0xf80u) : "memory"); }
#define ISSUET(pre, slot) ISSUE(pre, slot) TOUCH()
#define SLACK "s_nop 7\n\t"
#define NOSYNC "s_nop 0\n\t"
#define SYNC(n) "s_waitcnt vmcnt(" #n ")\n\t" SLACK "s_barrier\n\t"
        const u32 sox0 = sox;
        ISSUE(NOSYNC, 0)
        __builtin_amdgcn_sched_barrier(0);
        const bool c3 = ((g ^ (kxw[4] >> 9)) & 1) == 0;
        const u32x4 rt = {kxw[0], kxw[1], (lead && !(c3 && nb == 3)) ? kxw[2] : 0u, kxw[3]};
        u32 vt = (vx & ~0xfffu) | ((sox0 + 256u + ((c3 && nb == 1) ? (lane & 1) * 128u : 0u)) & 0xf80u);
        const u32 sink = lds0 + RINGB + w * 256;
        const u32 lbx = (lds0 + w * 3072) ^ (lds0 + w * 3072 + 4 * BLKB);
        u32 adj = adjv;
        TOUCH()
        ISSUET(NOSYNC, 1) ISSUET(NOSYNC, 2) ISSUET(NOSYNC, 3)
        lb ^= lbx;
        ISSUET(SYNC(13), 0) ISSUET(SYNC(13), 1) ISSUET(SYNC(13), 2) ISSUET(SYNC(13), 3)
        sox = (sox + adj) & 0xf80u; adj ^= adjv; lb ^= lbx;
        ISSUE(SYNC(13), 0) ISSUE(SYNC(12), 1) ISSUE(SYNC(11), 2) ISSUE(SYNC(10), 3)
#pragma unroll 1
        for (int s = 0; s < 5; ++s) {
            sox = (sox + adj) & 0xf80u; adj ^= adjv; lb ^= lbx;
            ISSUE(SYNC(9), 0) ISSUE(SYNC(9), 1) ISSUE(SYNC(9), 2) ISSUE(SYNC(9), 3)
        }
        asm volatile(SYNC(9) SYNC(6) SYNC(3) SYNC(0) ::: "memory");
        if (w != 0) return;
        const int rank = p.rank;
        GAS char* myws = (GAS char*)p.myws;
        GAS u32* fl = (GAS u32*)((GAS char*)p.ws[dst] + WS_FLAGS) + rank * 32 + i;
        u32 call = __hip_atomic_load((GAS u32*)(myws + WS_CNT) + b, __ATOMIC_RELAXED, __HIP_MEMORY_SCOPE_AGENT) + 1;
        call = __builtin_amdgcn_readfirstlane(call);
        if (lane == 0) __hip_atomic_store((GAS u32*)(myws + WS_CNT) + b, call, __ATOMIC_RELAXED, __HIP_MEMORY_SCOPE_AGENT);
        asm volatile("" : "+s"(fl));
        BARRIER();
        if (dst != rank) {
            if (lane == 0) __hip_atomic_store(fl, call, __ATOMIC_RELAXED, __HIP_MEMORY_SCOPE_SYSTEM);
        } else if (i == 0) {
            GAS const char* fp = myws + WS_FLAGS + lane * 16;
            bool ok = (lane >> 3) == rank;
            const unsigned long long t0 = __builtin_amdgcn_s_memrealtime();
            for (;;) {
                const unsigned long long now = __builtin_amdgcn_s_memrealtime();
                if (!ok) {
                    u32x4 f;
                    asm volatile("global_load_dwordx4 %0, %1, off sc0 sc1\n\ts_waitcnt vmcnt(0)" : "=&v"(f) : "v"(fp) : "memory");
                    ok = f[0] >= call && f[1] >= call && f[2] >= call && f[3] >= call;
                }
                if (__all(ok)) break;
                if (now - t0 > DEADLINE_TICKS) {
                    if (lane == 0) {
                        __hip_atomic_store((GAS u32*)(myws + WS_TMO), call, __ATOMIC_RELAXED, __HIP_MEMORY_SCOPE_AGENT);
                        ((GAS unsigned short*)p.out[dst])[(size_t)rank * MLOC * NLOC] = 0x7FC0;
                    }
                    break;
                }
            }
        }
        return;
    }

    const int c = w - 4, mi = c & 1, nh = c >> 1;
    const int c16 = lane & 15, q = lane >> 4;
    GAS char* op = (GAS char*)p.out[dst] + ((size_t)(p.rank * MLOC + m0 + mi * 16 + c16) * NLOC + nl0 + nh * 32 + (q & 1) * 16 + (q >> 1) * 8) * 2;
    u32 xa[2], wa[2];
#pragma unroll
    for (int s = 0; s < 2; ++s) xa[s] = lds0 + (2 * mi + (c16 >> 3)) * 3072 + (c16 & 7) * 128 + (((4 * s + q) ^ (c16 >> 1)) * 16);
    {
        const int qq = c16 >> 2, pp = c16 & 3, x = ((qq >> 1) & 1) | ((q & 1) << 1);
#pragma unroll
        for (int t = 0; t < 2; ++t) wa[t] = lds0 + (q >> 1) * 3072 + 1024 + (q & 1) * 1024 + qq * 128 + (((2 * nh + t) ^ x) * 32) + pp * 8;
    }
    asm volatile("s_waitcnt lgkmcnt(0)" : "+v"(op), "+v"(xa[0]), "+v"(xa[1]), "+v"(wa[0]), "+v"(wa[1]));
    f32x4 acc[2] = {{0.f, 0.f, 0.f, 0.f}, {0.f, 0.f, 0.f, 0.f}};
    u32 cnt, dl;
    asm volatile(
        "s_mov_b32 %[cnt], 8\n\ts_mov_b32 %[dl], 49152\n\t"
        "s_barrier\n\t"
        "ds_read_b128 v[64:67], %[xa0] offset:0\n\t"
        "ds_read_b64_tr_b16 v[68:69], %[wa0] offset:0\n\tds_read_b64_tr_b16 v[70:71], %[wa0] offset:512\n\t"
        "ds_read_b64_tr_b16 v[72:73], %[wa1] offset:0\n\tds_read_b64_tr_b16 v[74:75], %[wa1] offset:512\n\t"
        "ds_read_b128 v[76:79], %[xa1] offset:0\n\t"
        "ds_read_b64_tr_b16 v[80:81], %[wa0] offset:6144\n\tds_read_b64_tr_b16 v[82:83], %[wa0] offset:6656\n\t"
        "ds_read_b64_tr_b16 v[84:85], %[wa1] offset:6144\n\tds_read_b64_tr_b16 v[86:87], %[wa1] offset:6656\n\t"
        "1:\n\t"
        "s_barrier\n\t"
        "ds_read_b128 v[88:91], %[xa0] offset:12288\n\t"
        "ds_read_b64_tr_b16 v[92:93], %[wa0] offset:12288\n\tds_read_b64_tr_b16 v[94:95], %[wa0] offset:12800\n\t"
        "ds_read_b64_tr_b16 v[96:97], %[wa1] offset:12288\n\tds_read_b64_tr_b16 v[98:99], %[wa1] offset:12800\n\t"
        "s_waitcnt lgkmcnt(10)\n\t"
        "v_mfma_f32_16x16x32_bf16 %[a0], v[68:71], v[64:67], %[a0]\n\tv_mfma_f32_16x16x32_bf16 %[a1], v[72:75], v[64:67], %[a1]\n\t"
        "ds_read_b128 v[100:103], %[xa1] offset:12288\n\t"
        "ds_read_b64_tr_b16 v[104:105], %[wa0] offset:18432\n\tds_read_b64_tr_b16 v[106:107], %[wa0] offset:18944\n\t"
        "ds_read_b64_tr_b16 v[108:109], %[wa1] offset:18432\n\tds_read_b64_tr_b16 v[110:111], %[wa1] offset:18944\n\t"
        "s_waitcnt lgkmcnt(10)\n\t"
        "v_mfma_f32_16x16x32_bf16 %[a0], v[80:83], v[76:79], %[a0]\n\tv_mfma_f32_16x16x32_bf16 %[a1], v[84:87], v[76:79], %[a1]\n\t"
        "s_barrier\n\t"
        "ds_read_b128 v[64:67], %[xa0] offset:24576\n\t"
        "ds_read_b64_tr_b16 v[68:69], %[wa0] offset:24576\n\tds_read_b64_tr_b16 v[70:71], %[wa0] offset:25088\n\t"
        "ds_read_b64_tr_b16 v[72:73], %[wa1] offset:24576\n\tds_read_b64_tr_b16 v[74:75], %[wa1] offset:25088\n\t"
        "s_waitcnt lgkmcnt(10)\n\t"
        "v_mfma_f32_16x16x32_bf16 %[a0], v[92:95], v[88:91], %[a0]\n\tv_mfma_f32_16x16x32_bf16 %[a1], v[96:99], v[88:91], %[a1]\n\t"
        "ds_read_b128 v[76:79], %[xa1] offset:24576\n\t"
        "ds_read_b64_tr_b16 v[80:81], %[wa0] offset:30720\n\tds_read_b64_tr_b16 v[82:83], %[wa0] offset:31232\n\t"
        "ds_read_b64_tr_b16 v[84:85], %[wa1] offset:30720\n\tds_read_b64_tr_b16 v[86:87], %[wa1] offset:31232\n\t"
        "s_waitcnt lgkmcnt(10)\n\t"
        "v_mfma_f32_16x16x32_bf16 %[a0], v[104:107], v[100:103], %[a0]\n\tv_mfma_f32_16x16x32_bf16 %[a1], v[108:111], v[100:103], %[a1]\n\t"
        "s_barrier\n\t"
        "ds_read_b128 v[88:91], %[xa0] offset:36864\n\t"
        "ds_read_b64_tr_b16 v[92:93], %[wa0] offset:36864\n\tds_read_b64_tr_b16 v[94:95], %[wa0] offset:37376\n\t"
        "ds_read_b64_tr_b16 v[96:97], %[wa1] offset:36864\n\tds_read_b64_tr_b16 v[98:99], %[wa1] offset:37376\n\t"
        "s_waitcnt lgkmcnt(10)\n\t"
        "v_mfma_f32_16x16x32_bf16 %[a0], v[68:71], v[64:67], %[a0]\n\tv_mfma_f32_16x16x32_bf16 %[a1], v[72:75], v[64:67], %[a1]\n\t"
        "ds_read_b128 v[100:103], %[xa1] offset:36864\n\t"
        "ds_read_b64_tr_b16 v[104:105], %[wa0] offset:43008\n\tds_read_b64_tr_b16 v[106:107], %[wa0] offset:43520\n\t"
        "ds_read_b64_tr_b16 v[108:109], %[wa1] offset:43008\n\tds_read_b64_tr_b16 v[110:111], %[wa1] offset:43520\n\t"
        "v_add_u32 %[xa0], %[dl], %[xa0]\n\tv_add_u32 %[xa1], %[dl], %[xa1]\n\tv_add_u32 %[wa0], %[dl], %[wa0]\n\tv_add_u32 %[wa1], %[dl], %[wa1]\n\t"
        "s_sub_u32 %[dl], 0, %[dl]\n\ts_sub_u32 %[cnt], %[cnt], 1\n\ts_cmp_eq_u32 %[cnt], 0\n\t"
        "s_waitcnt lgkmcnt(10)\n\t"
        "v_mfma_f32_16x16x32_bf16 %[a0], v[80:83], v[76:79], %[a0]\n\tv_mfma_f32_16x16x32_bf16 %[a1], v[84:87], v[76:79], %[a1]\n\t"
        "s_cbranch_scc1 2f\n\t"
        "s_barrier\n\t"
        "ds_read_b128 v[64:67], %[xa0] offset:0\n\t"
        "ds_read_b64_tr_b16 v[68:69], %[wa0] offset:0\n\tds_read_b64_tr_b16 v[70:71], %[wa0] offset:512\n\t"
        "ds_read_b64_tr_b16 v[72:73], %[wa1] offset:0\n\tds_read_b64_tr_b16 v[74:75], %[wa1] offset:512\n\t"
        "s_waitcnt lgkmcnt(10)\n\t"
        "v_mfma_f32_16x16x32_bf16 %[a0], v[92:95], v[88:91], %[a0]\n\tv_mfma_f32_16x16x32_bf16 %[a1], v[96:99], v[88:91], %[a1]\n\t"
        "ds_read_b128 v[76:79], %[xa1] offset:0\n\t"
        "ds_read_b64_tr_b16 v[80:81], %[wa0] offset:6144\n\tds_read_b64_tr_b16 v[82:83], %[wa0] offset:6656\n\t"
        "ds_read_b64_tr_b16 v[84:85], %[wa1] offset:6144\n\tds_read_b64_tr_b16 v[86:87], %[wa1] offset:6656\n\t"
        "s_waitcnt lgkmcnt(10)\n\t"
        "v_mfma_f32_16x16x32_bf16 %[a0], v[104:107], v[100:103], %[a0]\n\tv_mfma_f32_16x16x32_bf16 %[a1], v[108:111], v[100:103], %[a1]\n\t"
        "s_branch 1b\n\t"
        "2:\n\t"
        "s_waitcnt lgkmcnt(5)\n\t"
        "v_mfma_f32_16x16x32_bf16 %[a0], v[92:95], v[88:91], %[a0]\n\tv_mfma_f32_16x16x32_bf16 %[a1], v[96:99], v[88:91], %[a1]\n\t"
        "s_waitcnt lgkmcnt(0)\n\t"
        "v_mfma_f32_16x16x32_bf16 %[a0], v[104:107], v[100:103], %[a0]\n\tv_mfma_f32_16x16x32_bf16 %[a1], v[108:111], v[100:103], %[a1]\n\t"
        "s_nop 7"
        : [a0] "+v"(acc[0]), [a1] "+v"(acc[1]), [xa0] "+v"(xa[0]), [xa1] "+v"(xa[1]), [wa0] "+v"(wa[0]), [wa1] "+v"(wa[1]),
          [cnt] "=&s"(cnt), [dl] "=&s"(dl)
        :
        : "memory", "scc", "v64", "v65", "v66", "v67", "v68", "v69", "v70", "v71", "v72", "v73", "v74", "v75", "v76", "v77", "v78", "v79", "v80", "v81", "v82", "v83", "v84", "v85", "v86", "v87", "v88", "v89", "v90", "v91", "v92", "v93", "v94", "v95", "v96", "v97", "v98", "v99", "v100", "v101", "v102", "v103", "v104", "v105", "v106", "v107", "v108", "v109", "v110", "v111");
    {
        const auto s0 = __builtin_amdgcn_permlane16_swap(pack_bf16(acc[0][0], acc[0][1]), pack_bf16(acc[1][0], acc[1][1]), false, false);
        const auto s1 = __builtin_amdgcn_permlane16_swap(pack_bf16(acc[0][2], acc[0][3]), pack_bf16(acc[1][2], acc[1][3]), false, false);
        const u32x4 o = {s0[0], s1[0], s0[1], s1[1]};
        asm volatile("global_store_dwordx4 %0, %1, off sc0 sc1\n\ts_nop 1" :: "v"(op), "v"(o) : "memory");
    }
    wait_vm<0>();
    BARRIER();
}

extern "C" void kernel_launch(int rank, int nranks, ncclComm_t comm,
                              hipStream_t stream,
                              void* const* d_in, const int* in_sizes, int n_in,
                              void* d_out, int out_size,
                              const int* in_dtypes, int out_dtype,
                              void* const* d_out_all,
                              void* const* d_ws_all, size_t ws_bytes) {
    if (nranks != NR || (unsigned)rank >= (unsigned)NR || n_in < 2 || ws_bytes < WS_NEED ||
        in_sizes[0] != MLOC * KD || in_sizes[1] != KD * ND || out_size != NR * MLOC * NLOC) {
        fprintf(stderr, "kernel_launch: unexpected problem shape\n");
        return;
    }
    Params p;
    const unsigned long long ax = (unsigned long long)d_in[0], aw = (unsigned long long)d_in[1] - 1024;
    if ((ax >> 48) || (aw >> 48)) { fprintf(stderr, "kernel_launch: address beyond 48 bits\n"); return; }
    p.rx[0] = (u32)ax; p.rx[1] = (u32)(ax >> 32); p.rx[2] = MLOC * KD * 2; p.rx[3] = 0x00020000u;
    p.rw[0] = (u32)aw; p.rw[1] = (u32)(aw >> 32); p.rw[2] = KD * ND * 2 + 1024; p.rw[3] = 0x00020000u;
    for (int r = 0; r < NR; ++r) { p.out[r] = d_out_all[r]; p.ws[r] = d_ws_all[r]; }
    p.rank = rank; p.pad = 0;
    p.myws = d_ws_all[rank];
    hipLaunchKernelGGL(gemm_a2a, dim3(NWG), dim3(NTHR), 0, stream, p);
    HIP_CHECK(hipGetLastError());
}
```
